```python
import numpy as np

kernel_name = "gpu_gemm_a2a_m4096_k4096_n4096_bf16_silu"


ABSMAX_RTOL = 0.008

COMPUTE_DTYPE = "bfloat16"
EPILOGUE = "silu"
IN_QUANT = None
OUT_QUANT = None
IN_DTYPES = {"x": "bfloat16", "w_mat": "bfloat16"}
WORLD_SIZE = 8

def _to_bf16(x):
    u = np.ascontiguousarray(x, dtype=np.float32).view(np.uint32)
    return ((u + 0x7FFF + ((u >> 16) & 1)) & 0xFFFF0000).view(np.float32)

def setup_inputs():

    x = _to_bf16(np.random.randn(4096, 4096).astype(np.float32))
    w_mat = _to_bf16((np.random.randn(4096, 4096) / np.sqrt(4096)).astype(np.float32))
    return {"x": x, "w_mat": w_mat}

def _epilogue(y):
    return (y / (1.0 + np.exp(-np.clip(y, -60.0, 60.0)))).astype(np.float32)

def reference(x, w_mat):
    y = (x @ w_mat).astype(np.float32)
    return _epilogue(y)
```

```cpp
#include <hip/hip_runtime.h>
#include <rccl/rccl.h>
#include <cstdio>

typedef unsigned short u16;
typedef unsigned u32x4 __attribute__((ext_vector_type(4)));
typedef unsigned u32x2 __attribute__((ext_vector_type(2)));
typedef __bf16 bf16x4 __attribute__((ext_vector_type(4)));
typedef __bf16 bf16x8 __attribute__((ext_vector_type(8)));
typedef float f32x4 __attribute__((ext_vector_type(4)));
#define LAS __attribute__((address_space(3)))

constexpr int NR = 8;
constexpr int MLOC = 512;
constexpr int KD = 4096;
constexpr int ND = 4096;
constexpr int NLOC = 512;
constexpr int BM = 128, BN = 64, BK = 64;
constexpr int NSTAGE = 4;
constexpr int XBYTES = BM * BK * 2;
constexpr int WBYTES = BN * BK * 2;
constexpr int STAGE = XBYTES + WBYTES;
constexpr int LDS_BYTES = NSTAGE * STAGE;
constexpr int NWG = 256;

constexpr size_t WS_FLAGS = 0;
constexpr size_t WS_CNT = 4096;
constexpr size_t WS_TMO = 8192;
constexpr size_t WS_NEED = 16384;
constexpr unsigned long long DEADLINE_TICKS = 20000000ull;

struct Params {
  const u16* x; const u16* w;
  u16* out[NR]; unsigned* ws[NR];
  int rank; int nranks;
};

template <int N_> __device__ __forceinline__ void wait_vm() { asm volatile("s_waitcnt vmcnt(%0)" :: "i"(N_) : "memory"); }

#if defined(__HIP_DEVICE_COMPILE__)
__device__ __forceinline__ void dma16(__amdgpu_buffer_rsrc_t rsrc, unsigned voff, unsigned soff, unsigned ldsaddr) {
  asm volatile("s_mov_b32 m0, %3\n\ts_nop 0\n\tbuffer_load_dwordx4 %0, %1, %2 offen lds"
               :: "v"(voff), "s"(rsrc), "s"(soff), "s"(ldsaddr) : "memory");
}
#endif
__device__ __forceinline__ float silu_f(float y) {
  float c = __builtin_amdgcn_fmed3f(y, -60.0f, 60.0f);
  float e = __builtin_amdgcn_exp2f(c * -1.4426950408889634f);
  return y * __builtin_amdgcn_rcpf(1.0f + e);
}
__device__ __forceinline__ unsigned pack_bf16(float lo, float hi) {
  typedef __bf16 bf16x2 __attribute__((ext_vector_type(2)));
  bf16x2 v = {(__bf16)lo, (__bf16)hi};
  return __builtin_bit_cast(unsigned, v);
}

__global__ __launch_bounds__(256) void gemm_a2a_kernel(Params p) {
#if defined(__HIP_DEVICE_COMPILE__)
  extern __shared__ __attribute__((aligned(16))) char smem[];
  LAS char* lds = (LAS char*)smem;
  const int bid = blockIdx.x;
  const int dest = bid & 7, li = bid >> 3;
  const int tm = li & 3, tn = li >> 2;
  const int row0 = tm * BM, col0 = dest * NLOC + tn * BN;
  const int tid = threadIdx.x, lane = tid & 63;
  const int wid = __builtin_amdgcn_readfirstlane(tid >> 6);
  const int wm = wid >> 1, wn = wid & 1;

  auto xr = __builtin_amdgcn_make_buffer_rsrc((void*)p.x, 0, MLOC * KD * 2, 0x00020000);
  auto wr = __builtin_amdgcn_make_buffer_rsrc((void*)p.w, 0, KD * ND * 2, 0x00020000);

  unsigned xoff[4], woff[2];
#pragma unroll
  for (int i = 0; i < 4; ++i) {
    int r = (wid * 4 + i) * 8 + (lane >> 3);
    int ch = (lane & 7) ^ ((r >> 1) & 7);
    xoff[i] = (unsigned)((row0 + r) * (KD * 2) + ch * 16);
  }
#pragma unroll
  for (int i = 0; i < 2; ++i) {
    int k = (wid * 2 + i) * 8 + (lane >> 3);
    int h = ((k >> 1) & 1) | (((k >> 3) & 1) << 1);
    int ch = (lane & 7) ^ (h << 1);
    woff[i] = (unsigned)(k * (ND * 2) + col0 * 2 + ch * 16);
  }
  const unsigned lds0 = (unsigned)(size_t)lds;
  auto issue = [&](int s) {
    unsigned base = lds0 + (unsigned)(s & (NSTAGE - 1)) * STAGE;
    unsigned sx = (unsigned)s * (BK * 2), sw = (unsigned)s * (BK * ND * 2);
#pragma unroll
    for (int i = 0; i < 4; ++i) dma16(xr, xoff[i], sx, base + (wid * 4 + i) * 1024);
#pragma unroll
    for (int i = 0; i < 2; ++i) dma16(wr, woff[i], sw, base + XBYTES + (wid * 2 + i) * 1024);
  };

  const int fr = lane & 15, fg = lane >> 4;
  unsigned xrd[4];
#pragma unroll
  for (int mt = 0; mt < 4; ++mt) {
    int r = wm * 64 + mt * 16 + fr;
    xrd[mt] = (unsigned)(r * 128 + ((fg ^ ((r >> 1) & 7)) * 16));
  }
  const int q = (lane & 15) >> 2, pp = lane & 3;
  unsigned wrd[2][2];
#pragma unroll
  for (int nt = 0; nt < 2; ++nt)
#pragma unroll
    for (int t = 0; t < 2; ++t) {
      int k = 8 * fg + 4 * t + q;
      int h = ((k >> 1) & 1) | (((k >> 3) & 1) << 1);
      int j = wn * 2 + nt;
      int ch = (2 * j + (pp >> 1)) ^ (h << 1);
      wrd[nt][t] = (unsigned)(XBYTES + k * 128 + ch * 16 + (pp & 1) * 8);
    }

  f32x4 acc[4][2];
#pragma unroll
  for (int a = 0; a < 4; ++a)
#pragma unroll
    for (int b = 0; b < 2; ++b) acc[a][b] = (f32x4){0.f, 0.f, 0.f, 0.f};

  constexpr int KS = KD / BK;
  issue(0); issue(1); issue(2);
  for (int s = 0; s < KS; ++s) {
    if (s + 2 < KS) wait_vm<12>(); else if (s + 1 < KS) wait_vm<6>(); else wait_vm<0>();
    __builtin_amdgcn_s_barrier();
    asm volatile("" ::: "memory");
    if (s + 3 < KS) issue(s + 3);
    LAS char* base = lds + (s & (NSTAGE - 1)) * STAGE;
#pragma unroll
    for (int ks = 0; ks < 2; ++ks) {
      bf16x8 xf[4], wf[2];
#pragma unroll
      for (int nt = 0; nt < 2; ++nt) {
        bf16x4 lo = __builtin_amdgcn_ds_read_tr16_b64_v4bf16((LAS bf16x4*)(base + wrd[nt][0] + ks * 4096));
        bf16x4 hi = __builtin_amdgcn_ds_read_tr16_b64_v4bf16((LAS bf16x4*)(base + wrd[nt][1] + ks * 4096));
        wf[nt] = __builtin_shufflevector(lo, hi, 0, 1, 2, 3, 4, 5, 6, 7);
      }
#pragma unroll
      for (int mt = 0; mt < 4; ++mt) xf[mt] = *(LAS bf16x8*)(base + (xrd[mt] ^ (ks * 64)));
#pragma unroll
      for (int mt = 0; mt < 4; ++mt)
#pragma unroll
        for (int nt = 0; nt < 2; ++nt)
          acc[mt][nt] = __builtin_amdgcn_mfma_f32_16x16x32_bf16(wf[nt], xf[mt], acc[mt][nt], 0, 0, 0);
    }
    asm volatile("s_waitcnt lgkmcnt(0)" ::: "memory");
  }
  __syncthreads();

#pragma unroll
  for (int mt = 0; mt < 4; ++mt)
#pragma unroll
    for (int nt = 0; nt < 2; ++nt) {
      f32x4 v = acc[mt][nt];
      u32x2 w;
      w.x = pack_bf16(silu_f(v[0]), silu_f(v[1]));
      w.y = pack_bf16(silu_f(v[2]), silu_f(v[3]));
      int m = wm * 64 + mt * 16 + fr;
      int n = wn * 32 + nt * 16 + fg * 4;
      int ch = (n >> 3) ^ (m & 7);
      *(LAS u32x2*)(lds + m * 128 + ch * 16 + (n & 4) * 2) = w;
    }
  __syncthreads();
  {
    u16* obase = p.out[dest] + (size_t)(p.rank * MLOC + row0) * NLOC + tn * BN;
#pragma unroll
    for (int i = 0; i < 4; ++i) {
      int m = (wid * 4 + i) * 8 + (lane >> 3);
      int c = lane & 7;
      u32x4 v = *(LAS u32x4*)(lds + m * 128 + ((c ^ (m & 7)) * 16));
      u16* dst = obase + (size_t)m * NLOC + c * 8;
      asm volatile("global_store_dwordx4 %0, %1, off sc0 sc1\n\ts_nop 1" :: "v"(dst), "v"(v) : "memory");
    }
  }
  unsigned* myws = p.ws[p.rank];
  unsigned call = 0;
  if (tid == 0) call = __hip_atomic_load(myws + WS_CNT / 4 + bid, __ATOMIC_RELAXED, __HIP_MEMORY_SCOPE_AGENT) + 1;
  asm volatile("s_waitcnt vmcnt(0)" ::: "memory");
  __syncthreads();
  if (wid == 0) {
    call = __builtin_amdgcn_readfirstlane(call);
    if (lane == 0) {
      __hip_atomic_store(p.ws[dest] + WS_FLAGS / 4 + p.rank * 32 + li, call, __ATOMIC_RELAXED, __HIP_MEMORY_SCOPE_SYSTEM);
      __hip_atomic_store(myws + WS_CNT / 4 + bid, call, __ATOMIC_RELAXED, __HIP_MEMORY_SCOPE_AGENT);
    }
    if (bid == 0) {
      unsigned* tmo = myws + WS_TMO / 4;
      unsigned* fl = myws + WS_FLAGS / 4 + lane * 4;
      unsigned long long t0 = __builtin_amdgcn_s_memrealtime();
      bool bad = false;
      for (;;) {
        unsigned a = __hip_atomic_load(fl + 0, __ATOMIC_RELAXED, __HIP_MEMORY_SCOPE_SYSTEM);
        unsigned b = __hip_atomic_load(fl + 1, __ATOMIC_RELAXED, __HIP_MEMORY_SCOPE_SYSTEM);
        unsigned c = __hip_atomic_load(fl + 2, __ATOMIC_RELAXED, __HIP_MEMORY_SCOPE_SYSTEM);
        unsigned d = __hip_atomic_load(fl + 3, __ATOMIC_RELAXED, __HIP_MEMORY_SCOPE_SYSTEM);
        bool ok = (a >= call) & (b >= call) & (c >= call) & (d >= call);
        if (__all(ok)) break;
        if (__builtin_amdgcn_s_memrealtime() - t0 > DEADLINE_TICKS) { bad = true; break; }
        __builtin_amdgcn_s_sleep(1);
      }
      if (bad && lane == 0) {
        __hip_atomic_store(tmo, call, __ATOMIC_RELAXED, __HIP_MEMORY_SCOPE_AGENT);
        u16* o = p.out[p.rank] + (size_t)(p.rank * MLOC) * NLOC;
        __hip_atomic_store(o, (u16)0x7fc0, __ATOMIC_RELAXED, __HIP_MEMORY_SCOPE_SYSTEM);
      }
    }
  }
#endif
}

extern "C" void kernel_launch(int rank, int nranks, ncclComm_t comm, hipStream_t stream,
                              void* const* d_in, const int* in_sizes, int n_in,
                              void* d_out, int out_size, const int* in_dtypes, int out_dtype,
                              void* const* d_out_all, void* const* d_ws_all, size_t ws_bytes) {
  if (nranks != NR || n_in < 2 || in_sizes[0] != MLOC * KD || in_sizes[1] != KD * ND || out_size != ND * NLOC || ws_bytes < WS_NEED) {
    fprintf(stderr, "unexpected problem shape\n");
    return;
  }
  static bool init = false;
  if (!init) {
    hipError_t e = hipFuncSetAttribute((const void*)gemm_a2a_kernel, hipFuncAttributeMaxDynamicSharedMemorySize, LDS_BYTES);
    if (e != hipSuccess) { fprintf(stderr, "hipFuncSetAttribute: %s\n", hipGetErrorString(e)); return; }
    init = true;
  }
  Params p;
  p.x = (const u16*)d_in[0]; p.w = (const u16*)d_in[1];
  for (int i = 0; i < NR; ++i) { p.out[i] = (u16*)d_out_all[i]; p.ws[i] = (unsigned*)d_ws_all[i]; }
  p.rank = rank; p.nranks = nranks;
  gemm_a2a_kernel<<<NWG, 256, LDS_BYTES, stream>>>(p);
}
```

```cpp
#include <hip/hip_runtime.h>
#include <rccl/rccl.h>
#include <cstdio>
#include <cstddef>

typedef unsigned short u16;
typedef unsigned u32x4 __attribute__((ext_vector_type(4)));

#define S2_VAR 0
#define S2_ASM \
  "s_mov_b32 s73, 0x42ad1f97\n" \
  "s_mov_b32 s71, %[obase]\n" \
  "s_load_dwordx2 s[92:93], %[optr], 0x0\n" \
  "s_bfe_u32 s84, %[perm], 196608\n" \
  "s_lshl_b32 s84, s84, 10\n" \
  "s_bfe_u32 s85, %[perm], 196612\n" \
  "s_lshl_b32 s85, s85, 10\n" \
  "s_bfe_u32 s86, %[perm], 196616\n" \
  "s_lshl_b32 s86, s86, 10\n" \
  "s_bfe_u32 s87, %[perm], 196620\n" \
  "s_lshl_b32 s87, s87, 10\n" \
  "s_bfe_u32 s88, %[perm], 196624\n" \
  "s_lshl_b32 s88, s88, 10\n" \
  "s_bfe_u32 s89, %[perm], 196628\n" \
  "s_lshl_b32 s89, s89, 10\n" \
  "s_bfe_u32 s90, %[perm], 196632\n" \
  "s_lshl_b32 s90, s90, 10\n" \
  "s_bfe_u32 s91, %[perm], 196636\n" \
  "s_lshl_b32 s91, s91, 10\n" \
  "s_add_u32 s65, %[xrow], s84\n" \
  "s_add_u32 s66, s65, 64512\n" \
  "s_add_u32 m0, %[w4k], 0\n" \
  "s_nop 0\n" \
  "buffer_load_dwordx4 %[A1], %[xrs], s65 offen lds\n" \
  "buffer_load_dwordx4 %[A1], %[xrs], s66 offen offset:1024 lds\n" \
  "buffer_load_dwordx4 %[A2], %[xrs], s65 offen offset:2048 lds\n" \
  "buffer_load_dwordx4 %[A2], %[xrs], s66 offen offset:3072 lds\n" \
  "s_lshl_b32 s74, s84, 12\n" \
  "s_add_u32 s67, s74, 0\n" \
  "s_add_u32 s68, s74, 31744\n" \
  "s_add_u32 s69, s74, 129024\n" \
  "s_add_u32 s70, s74, 160768\n" \
  "s_add_u32 m0, %[w4k], 16384\n" \
  "s_nop 0\n" \
  "buffer_load_dwordx4 %[A3], %[wrs], s67 offen offset:0 lds\n" \
  "buffer_load_dwordx4 %[A3], %[wrs], s68 offen offset:1024 lds\n" \
  "buffer_load_dwordx4 %[A3], %[wrs], s69 offen offset:2048 lds\n" \
  "buffer_load_dwordx4 %[A3], %[wrs], s70 offen offset:3072 lds\n" \
  "s_lshl_b32 s74, s84, 12\n" \
  "s_add_u32 s67, s74, 262144\n" \
  "s_add_u32 s68, s74, 293888\n" \
  "s_add_u32 s69, s74, 391168\n" \
  "s_add_u32 s70, s74, 422912\n" \
  "s_add_u32 m0, %[w4k], 32768\n" \
  "s_nop 0\n" \
  "buffer_load_dwordx4 %[A3], %[wrs], s67 offen offset:0 lds\n" \
  "buffer_load_dwordx4 %[A3], %[wrs], s68 offen offset:1024 lds\n" \
  "buffer_load_dwordx4 %[A3], %[wrs], s69 offen offset:2048 lds\n" \
  "buffer_load_dwordx4 %[A3], %[wrs], s70 offen offset:3072 lds\n" \
  "s_waitcnt vmcnt(8)\n" \
  "ds_read_b128 v[48:51], %[vX0] offset:0\n" \
  "ds_read_b128 v[52:55], %[vX1] offset:0\n" \
  "ds_read_b128 v[56:59], %[vX0] offset:2048\n" \
  "ds_read_b128 v[60:63], %[vX1] offset:2048\n" \
  "s_lshl_b32 s74, s84, 12\n" \
  "s_add_u32 s67, s74, 524288\n" \
  "s_add_u32 s68, s74, 556032\n" \
  "s_add_u32 s69, s74, 653312\n" \
  "s_add_u32 s70, s74, 685056\n" \
  "s_add_u32 m0, %[w4k], 49152\n" \
  "s_nop 0\n" \
  "buffer_load_dwordx4 %[A3], %[wrs], s67 offen offset:0 lds\n" \
  "buffer_load_dwordx4 %[A3], %[wrs], s68 offen offset:1024 lds\n" \
  "buffer_load_dwordx4 %[A3], %[wrs], s69 offen offset:2048 lds\n" \
  "buffer_load_dwordx4 %[A3], %[wrs], s70 offen offset:3072 lds\n" \
  "s_waitcnt vmcnt(8)\n" \
  "ds_read_b64_tr_b16 a[0:1], %[vT0] offset:16384\n" \
  "ds_read_b64_tr_b16 a[2:3], %[vT0] offset:17408\n" \
  "ds_read_b64_tr_b16 a[4:5], %[vT1] offset:16384\n" \
  "ds_read_b64_tr_b16 a[6:7], %[vT1] offset:17408\n" \
  "ds_read_b64_tr_b16 a[8:9], %[vT2] offset:16384\n" \
  "ds_read_b64_tr_b16 a[10:11], %[vT2] offset:17408\n" \
  "ds_read_b64_tr_b16 a[12:13], %[vT3] offset:16384\n" \
  "ds_read_b64_tr_b16 a[14:15], %[vT3] offset:17408\n" \
  "s_waitcnt lgkmcnt(8)\n" \
  "s_lshl_b32 s74, s84, 12\n" \
  "s_add_u32 s67, s74, 786432\n" \
  "s_add_u32 s68, s74, 818176\n" \
  "s_add_u32 s69, s74, 915456\n" \
  "s_add_u32 s70, s74, 947200\n" \
  "s_add_u32 m0, %[w4k], 0\n" \
  "s_nop 0\n" \
  "buffer_load_dwordx4 %[A3], %[wrs], s67 offen offset:0 lds\n" \
  "buffer_load_dwordx4 %[A3], %[wrs], s68 offen offset:1024 lds\n" \
  "buffer_load_dwordx4 %[A3], %[wrs], s69 offen offset:2048 lds\n" \
  "buffer_load_dwordx4 %[A3], %[wrs], s70 offen offset:3072 lds\n" \
  "s_waitcnt vmcnt(8)\n" \
  "ds_read_b64_tr_b16 a[16:17], %[vT0] offset:32768\n" \
  "ds_read_b64_tr_b16 a[18:19], %[vT0] offset:33792\n" \
  "ds_read_b64_tr_b16 a[20:21], %[vT1] offset:32768\n" \
  "ds_read_b64_tr_b16 a[22:23], %[vT1] offset:33792\n" \
  "ds_read_b64_tr_b16 a[24:25], %[vT2] offset:32768\n" \
  "ds_read_b64_tr_b16 a[26:27], %[vT2] offset:33792\n" \
  "ds_read_b64_tr_b16 a[28:29], %[vT3] offset:32768\n" \
  "ds_read_b64_tr_b16 a[30:31], %[vT3] offset:33792\n" \
  "s_waitcnt lgkmcnt(8)\n" \
  "s_add_u32 s65, %[xrow], s85\n" \
  "s_add_u32 s66, s65, 64512\n" \
  "s_add_u32 m0, %[w4k], 16384\n" \
  "s_nop 0\n" \
  "buffer_load_dwordx4 %[A1], %[xrs], s65 offen lds\n" \
  "buffer_load_dwordx4 %[A1], %[xrs], s66 offen offset:1024 lds\n" \
  "buffer_load_dwordx4 %[A2], %[xrs], s65 offen offset:2048 lds\n" \
  "buffer_load_dwordx4 %[A2], %[xrs], s66 offen offset:3072 lds\n" \
  "v_mfma_f32_16x16x32_bf16 v[32:35], a[0:3], v[48:51], 0\n" \
  "v_mfma_f32_16x16x32_bf16 v[36:39], a[4:7], v[48:51], 0\n" \
  "v_mfma_f32_16x16x32_bf16 v[40:43], a[8:11], v[48:51], 0\n" \
  "v_mfma_f32_16x16x32_bf16 v[44:47], a[12:15], v[48:51], 0\n" \
  "s_waitcnt vmcnt(8)\n" \
  "ds_read_b64_tr_b16 a[32:33], %[vT0] offset:49152\n" \
  "ds_read_b64_tr_b16 a[34:35], %[vT0] offset:50176\n" \
  "ds_read_b64_tr_b16 a[36:37], %[vT1] offset:49152\n" \
  "ds_read_b64_tr_b16 a[38:39], %[vT1] offset:50176\n" \
  "ds_read_b64_tr_b16 a[40:41], %[vT2] offset:49152\n" \
  "ds_read_b64_tr_b16 a[42:43], %[vT2] offset:50176\n" \
  "ds_read_b64_tr_b16 a[44:45], %[vT3] offset:49152\n" \
  "ds_read_b64_tr_b16 a[46:47], %[vT3] offset:50176\n" \
  "s_waitcnt lgkmcnt(8)\n" \
  "s_lshl_b32 s74, s85, 12\n" \
  "s_add_u32 s67, s74, 0\n" \
  "s_add_u32 s68, s74, 31744\n" \
  "s_add_u32 s69, s74, 129024\n" \
  "s_add_u32 s70, s74, 160768\n" \
  "s_add_u32 m0, %[w4k], 32768\n" \
  "s_nop 0\n" \
  "buffer_load_dwordx4 %[A3], %[wrs], s67 offen offset:0 lds\n" \
  "buffer_load_dwordx4 %[A3], %[wrs], s68 offen offset:1024 lds\n" \
  "buffer_load_dwordx4 %[A3], %[wrs], s69 offen offset:2048 lds\n" \
  "buffer_load_dwordx4 %[A3], %[wrs], s70 offen offset:3072 lds\n" \
  "v_mfma_f32_16x16x32_bf16 v[32:35], a[16:19], v[52:55], v[32:35]\n" \
  "v_mfma_f32_16x16x32_bf16 v[36:39], a[20:23], v[52:55], v[36:39]\n" \
  "v_mfma_f32_16x16x32_bf16 v[40:43], a[24:27], v[52:55], v[40:43]\n" \
  "v_mfma_f32_16x16x32_bf16 v[44:47], a[28:31], v[52:55], v[44:47]\n" \
  "s_waitcnt vmcnt(8)\n" \
  "ds_read_b64_tr_b16 a[48:49], %[vT0] offset:0\n" \
  "ds_read_b64_tr_b16 a[50:51], %[vT0] offset:1024\n" \
  "ds_read_b64_tr_b16 a[52:53], %[vT1] offset:0\n" \
  "ds_read_b64_tr_b16 a[54:55], %[vT1] offset:1024\n" \
  "ds_read_b64_tr_b16 a[56:57], %[vT2] offset:0\n" \
  "ds_read_b64_tr_b16 a[58:59], %[vT2] offset:1024\n" \
  "ds_read_b64_tr_b16 a[60:61], %[vT3] offset:0\n" \
  "ds_read_b64_tr_b16 a[62:63], %[vT3] offset:1024\n" \
  "s_waitcnt lgkmcnt(8)\n" \
  "s_lshl_b32 s74, s85, 12\n" \
  "s_add_u32 s67, s74, 262144\n" \
  "s_add_u32 s68, s74, 293888\n" \
  "s_add_u32 s69, s74, 391168\n" \
  "s_add_u32 s70, s74, 422912\n" \
  "s_add_u32 m0, %[w4k], 49152\n" \
  "s_nop 0\n" \
  "buffer_load_dwordx4 %[A3], %[wrs], s67 offen offset:0 lds\n" \
  "buffer_load_dwordx4 %[A3], %[wrs], s68 offen offset:1024 lds\n" \
  "buffer_load_dwordx4 %[A3], %[wrs], s69 offen offset:2048 lds\n" \
  "buffer_load_dwordx4 %[A3], %[wrs], s70 offen offset:3072 lds\n" \
  "v_mfma_f32_16x16x32_bf16 v[32:35], a[32:35], v[56:59], v[32:35]\n" \
  "v_mfma_f32_16x16x32_bf16 v[36:39], a[36:39], v[56:59], v[36:39]\n" \
  "v_mfma_f32_16x16x32_bf16 v[40:43], a[40:43], v[56:59], v[40:43]\n" \
  "v_mfma_f32_16x16x32_bf16 v[44:47], a[44:47], v[56:59], v[44:47]\n" \
  "s_waitcnt vmcnt(8)\n" \
  "s_waitcnt lgkmcnt(0)\n" \
  "v_mfma_f32_16x16x32_bf16 v[32:35], a[48:51], v[60:63], v[32:35]\n" \
  "v_mfma_f32_16x16x32_bf16 v[36:39], a[52:55], v[60:63], v[36:39]\n" \
  "v_mfma_f32_16x16x32_bf16 v[40:43], a[56:59], v[60:63], v[40:43]\n" \
  "v_mfma_f32_16x16x32_bf16 v[44:47], a[60:63], v[60:63], v[44:47]\n" \
  "ds_read_b128 v[48:51], %[vX0] offset:16384\n" \
  "ds_read_b128 v[52:55], %[vX1] offset:16384\n" \
  "ds_read_b128 v[56:59], %[vX0] offset:18432\n" \
  "ds_read_b128 v[60:63], %[vX1] offset:18432\n" \
  "s_lshl_b32 s74, s85, 12\n" \
  "s_add_u32 s67, s74, 524288\n" \
  "s_add_u32 s68, s74, 556032\n" \
  "s_add_u32 s69, s74, 653312\n" \
  "s_add_u32 s70, s74, 685056\n" \
  "s_add_u32 m0, %[w4k], 0\n" \
  "s_nop 0\n" \
  "buffer_load_dwordx4 %[A3], %[wrs], s67 offen offset:0 lds\n" \
  "buffer_load_dwordx4 %[A3], %[wrs], s68 offen offset:1024 lds\n" \
  "buffer_load_dwordx4 %[A3], %[wrs], s69 offen offset:2048 lds\n" \
  "buffer_load_dwordx4 %[A3], %[wrs], s70 offen offset:3072 lds\n" \
  "s_waitcnt vmcnt(8)\n" \
  "ds_read_b64_tr_b16 a[64:65], %[vT0] offset:32768\n" \
  "ds_read_b64_tr_b16 a[66:67], %[vT0] offset:33792\n" \
  "ds_read_b64_tr_b16 a[68:69], %[vT1] offset:32768\n" \
  "ds_read_b64_tr_b16 a[70:71], %[vT1] offset:33792\n" \
  "ds_read_b64_tr_b16 a[72:73], %[vT2] offset:32768\n" \
  "ds_read_b64_tr_b16 a[74:75], %[vT2] offset:33792\n" \
  "ds_read_b64_tr_b16 a[76:77], %[vT3] offset:32768\n" \
  "ds_read_b64_tr_b16 a[78:79], %[vT3] offset:33792\n" \
  "s_waitcnt lgkmcnt(8)\n" \
  "s_lshl_b32 s74, s85, 12\n" \
  "s_add_u32 s67, s74, 786432\n" \
  "s_add_u32 s68, s74, 818176\n" \
  "s_add_u32 s69, s74, 915456\n" \
  "s_add_u32 s70, s74, 947200\n" \
  "s_add_u32 m0, %[w4k], 16384\n" \
  "s_nop 0\n" \
  "buffer_load_dwordx4 %[A3], %[wrs], s67 offen offset:0 lds\n" \
  "buffer_load_dwordx4 %[A3], %[wrs], s68 offen offset:1024 lds\n" \
  "buffer_load_dwordx4 %[A3], %[wrs], s69 offen offset:2048 lds\n" \
  "buffer_load_dwordx4 %[A3], %[wrs], s70 offen offset:3072 lds\n" \
  "s_waitcnt vmcnt(8)\n" \
  "ds_read_b64_tr_b16 a[80:81], %[vT0] offset:49152\n" \
  "ds_read_b64_tr_b16 a[82:83], %[vT0] offset:50176\n" \
  "ds_read_b64_tr_b16 a[84:85], %[vT1] offset:49152\n" \
  "ds_read_b64_tr_b16 a[86:87], %[vT1] offset:50176\n" \
  "ds_read_b64_tr_b16 a[88:89], %[vT2] offset:49152\n" \
  "ds_read_b64_tr_b16 a[90:91], %[vT2] offset:50176\n" \
  "ds_read_b64_tr_b16 a[92:93], %[vT3] offset:49152\n" \
  "ds_read_b64_tr_b16 a[94:95], %[vT3] offset:50176\n" \
  "s_waitcnt lgkmcnt(8)\n" \
  "s_add_u32 s65, %[xrow], s86\n" \
  "s_add_u32 s66, s65, 64512\n" \
  "s_add_u32 m0, %[w4k], 32768\n" \
  "s_nop 0\n" \
  "buffer_load_dwordx4 %[A1], %[xrs], s65 offen lds\n" \
  "buffer_load_dwordx4 %[A1], %[xrs], s66 offen offset:1024 lds\n" \
  "buffer_load_dwordx4 %[A2], %[xrs], s65 offen offset:2048 lds\n" \
  "buffer_load_dwordx4 %[A2], %[xrs], s66 offen offset:3072 lds\n" \
  "v_mfma_f32_16x16x32_bf16 v[32:35], a[64:67], v[48:51], v[32:35]\n" \
  "v_mfma_f32_16x16x32_bf16 v[36:39], a[68:71], v[48:51], v[36:39]\n" \
  "v_mfma_f32_16x16x32_bf16 v[40:43], a[72:75], v[48:51], v[40:43]\n" \
  "v_mfma_f32_16x16x32_bf16 v[44:47], a[76:79], v[48:51], v[44:47]\n" \
  "s_waitcnt vmcnt(8)\n" \
  "ds_read_b64_tr_b16 a[96:97], %[vT0] offset:0\n" \
  "ds_read_b64_tr_b16 a[98:99], %[vT0] offset:1024\n" \
  "ds_read_b64_tr_b16 a[100:101], %[vT1] offset:0\n" \
  "ds_read_b64_tr_b16 a[102:103], %[vT1] offset:1024\n" \
  "ds_read_b64_tr_b16 a[104:105], %[vT2] offset:0\n" \
  "ds_read_b64_tr_b16 a[106:107], %[vT2] offset:1024\n" \
  "ds_read_b64_tr_b16 a[108:109], %[vT3] offset:0\n" \
  "ds_read_b64_tr_b16 a[110:111], %[vT3] offset:1024\n" \
  "s_waitcnt lgkmcnt(8)\n" \
  "s_lshl_b32 s74, s86, 12\n" \
  "s_add_u32 s67, s74, 0\n" \
  "s_add_u32 s68, s74, 31744\n" \
  "s_add_u32 s69, s74, 129024\n" \
  "s_add_u32 s70, s74, 160768\n" \
  "s_add_u32 m0, %[w4k], 49152\n" \
  "s_nop 0\n" \
  "buffer_load_dwordx4 %[A3], %[wrs], s67 offen offset:0 lds\n" \
  "buffer_load_dwordx4 %[A3], %[wrs], s68 offen offset:1024 lds\n" \
  "buffer_load_dwordx4 %[A3], %[wrs], s69 offen offset:2048 lds\n" \
  "buffer_load_dwordx4 %[A3], %[wrs], s70 offen offset:3072 lds\n" \
  "v_mfma_f32_16x16x32_bf16 v[32:35], a[80:83], v[52:55], v[32:35]\n" \
  "v_mfma_f32_16x16x32_bf16 v[36:39], a[84:87], v[52:55], v[36:39]\n" \
  "v_mfma_f32_16x16x32_bf16 v[40:43], a[88:91], v[52:55], v[40:43]\n" \
  "v_mfma_f32_16x16x32_bf16 v[44:47], a[92:95], v[52:55], v[44:47]\n" \
  "s_waitcnt vmcnt(8)\n" \
  "ds_read_b64_tr_b16 a[112:113], %[vT0] offset:16384\n" \
  "ds_read_b64_tr_b16 a[114:115], %[vT0] offset:17408\n" \
  "ds_read_b64_tr_b16 a[116:117], %[vT1] offset:16384\n" \
  "ds_read_b64_tr_b16 a[118:119], %[vT1] offset:17408\n" \
  "ds_read_b64_tr_b16 a[120:121], %[vT2] offset:16384\n" \
  "ds_read_b64_tr_b16 a[122:123], %[vT2] offset:17408\n" \
  "ds_read_b64_tr_b16 a[124:125], %[vT3] offset:16384\n" \
  "ds_read_b64_tr_b16 a[126:127], %[vT3] offset:17408\n" \
  "s_waitcnt lgkmcnt(8)\n" \
  "s_lshl_b32 s74, s86, 12\n" \
  "s_add_u32 s67, s74, 262144\n" \
  "s_add_u32 s68, s74, 293888\n" \
  "s_add_u32 s69, s74, 391168\n" \
  "s_add_u32 s70, s74, 422912\n" \
  "s_add_u32 m0, %[w4k], 0\n" \
  "s_nop 0\n" \
  "buffer_load_dwordx4 %[A3], %[wrs], s67 offen offset:0 lds\n" \
  "buffer_load_dwordx4 %[A3], %[wrs], s68 offen offset:1024 lds\n" \
  "buffer_load_dwordx4 %[A3], %[wrs], s69 offen offset:2048 lds\n" \
  "buffer_load_dwordx4 %[A3], %[wrs], s70 offen offset:3072 lds\n" \
  "v_mfma_f32_16x16x32_bf16 v[32:35], a[96:99], v[56:59], v[32:35]\n" \
  "v_mfma_f32_16x16x32_bf16 v[36:39], a[100:103], v[56:59], v[36:39]\n" \
  "v_mfma_f32_16x16x32_bf16 v[40:43], a[104:107], v[56:59], v[40:43]\n" \
  "v_mfma_f32_16x16x32_bf16 v[44:47], a[108:111], v[56:59], v[44:47]\n" \
  "s_waitcnt vmcnt(8)\n" \
  "s_waitcnt lgkmcnt(0)\n" \
  "v_mfma_f32_16x16x32_bf16 v[32:35], a[112:115], v[60:63], v[32:35]\n" \
  "v_mfma_f32_16x16x32_bf16 v[36:39], a[116:119], v[60:63], v[36:39]\n" \
  "v_mfma_f32_16x16x32_bf16 v[40:43], a[120:123], v[60:63], v[40:43]\n" \
  "v_mfma_f32_16x16x32_bf16 v[44:47], a[124:127], v[60:63], v[44:47]\n" \
  "ds_read_b128 v[48:51], %[vX0] offset:32768\n" \
  "ds_read_b128 v[52:55], %[vX1] offset:32768\n" \
  "ds_read_b128 v[56:59], %[vX0] offset:34816\n" \
  "ds_read_b128 v[60:63], %[vX1] offset:34816\n" \
  "s_lshl_b32 s74, s86, 12\n" \
  "s_add_u32 s67, s74, 524288\n" \
  "s_add_u32 s68, s74, 556032\n" \
  "s_add_u32 s69, s74, 653312\n" \
  "s_add_u32 s70, s74, 685056\n" \
  "s_add_u32 m0, %[w4k], 16384\n" \
  "s_nop 0\n" \
  "buffer_load_dwordx4 %[A3], %[wrs], s67 offen offset:0 lds\n" \
  "buffer_load_dwordx4 %[A3], %[wrs], s68 offen offset:1024 lds\n" \
  "buffer_load_dwordx4 %[A3], %[wrs], s69 offen offset:2048 lds\n" \
  "buffer_load_dwordx4 %[A3], %[wrs], s70 offen offset:3072 lds\n" \
  "s_waitcnt vmcnt(8)\n" \
  "ds_read_b64_tr_b16 a[128:129], %[vT0] offset:49152\n" \
  "ds_read_b64_tr_b16 a[130:131], %[vT0] offset:50176\n" \
  "ds_read_b64_tr_b16 a[132:133], %[vT1] offset:49152\n" \
  "ds_read_b64_tr_b16 a[134:135], %[vT1] offset:50176\n" \
  "ds_read_b64_tr_b16 a[136:137], %[vT2] offset:49152\n" \
  "ds_read_b64_tr_b16 a[138:139], %[vT2] offset:50176\n" \
  "ds_read_b64_tr_b16 a[140:141], %[vT3] offset:49152\n" \
  "ds_read_b64_tr_b16 a[142:143], %[vT3] offset:50176\n" \
  "s_waitcnt lgkmcnt(8)\n" \
  "s_lshl_b32 s74, s86, 12\n" \
  "s_add_u32 s67, s74, 786432\n" \
  "s_add_u32 s68, s74, 818176\n" \
  "s_add_u32 s69, s74, 915456\n" \
  "s_add_u32 s70, s74, 947200\n" \
  "s_add_u32 m0, %[w4k], 32768\n" \
  "s_nop 0\n" \
  "buffer_load_dwordx4 %[A3], %[wrs], s67 offen offset:0 lds\n" \
  "buffer_load_dwordx4 %[A3], %[wrs], s68 offen offset:1024 lds\n" \
  "buffer_load_dwordx4 %[A3], %[wrs], s69 offen offset:2048 lds\n" \
  "buffer_load_dwordx4 %[A3], %[wrs], s70 offen offset:3072 lds\n" \
  "s_waitcnt vmcnt(8)\n" \
  "ds_read_b64_tr_b16 a[144:145], %[vT0] offset:0\n" \
  "ds_read_b64_tr_b16 a[146:147], %[vT0] offset:1024\n" \
  "ds_read_b64_tr_b16 a[148:149], %[vT1] offset:0\n" \
  "ds_read_b64_tr_b16 a[150:151], %[vT1] offset:1024\n" \
  "ds_read_b64_tr_b16 a[152:153], %[vT2] offset:0\n" \
  "ds_read_b64_tr_b16 a[154:155], %[vT2] offset:1024\n" \
  "ds_read_b64_tr_b16 a[156:157], %[vT3] offset:0\n" \
  "ds_read_b64_tr_b16 a[158:159], %[vT3] offset:1024\n" \
  "s_waitcnt lgkmcnt(8)\n" \
  "s_add_u32 s65, %[xrow], s87\n" \
  "s_add_u32 s66, s65, 64512\n" \
  "s_add_u32 m0, %[w4k], 49152\n" \
  "s_nop 0\n" \
  "buffer_load_dwordx4 %[A1], %[xrs], s65 offen lds\n" \
  "buffer_load_dwordx4 %[A1], %[xrs], s66 offen offset:1024 lds\n" \
  "buffer_load_dwordx4 %[A2], %[xrs], s65 offen offset:2048 lds\n" \
  "buffer_load_dwordx4 %[A2], %[xrs], s66 offen offset:3072 lds\n" \
  "v_mfma_f32_16x16x32_bf16 v[32:35], a[128:131], v[48:51], v[32:35]\n" \
  "v_mfma_f32_16x16x32_bf16 v[36:39], a[132:135], v[48:51], v[36:39]\n" \
  "v_mfma_f32_16x16x32_bf16 v[40:43], a[136:139], v[48:51], v[40:43]\n" \
  "v_mfma_f32_16x16x32_bf16 v[44:47], a[140:143], v[48:51], v[44:47]\n" \
  "s_waitcnt vmcnt(8)\n" \
  "ds_read_b64_tr_b16 a[160:161], %[vT0] offset:16384\n" \
  "ds_read_b64_tr_b16 a[162:163], %[vT0] offset:17408\n" \
  "ds_read_b64_tr_b16 a[164:165], %[vT1] offset:16384\n" \
  "ds_read_b64_tr_b16 a[166:167], %[vT1] offset:17408\n" \
  "ds_read_b64_tr_b16 a[168:169], %[vT2] offset:16384\n" \
  "ds_read_b64_tr_b16 a[170:171], %[vT2] offset:17408\n" \
  "ds_read_b64_tr_b16 a[172:173], %[vT3] offset:16384\n" \
  "ds_read_b64_tr_b16 a[174:175], %[vT3] offset:17408\n" \
  "s_waitcnt lgkmcnt(8)\n" \
  "s_lshl_b32 s74, s87, 12\n" \
  "s_add_u32 s67, s74, 0\n" \
  "s_add_u32 s68, s74, 31744\n" \
  "s_add_u32 s69, s74, 129024\n" \
  "s_add_u32 s70, s74, 160768\n" \
  "s_add_u32 m0, %[w4k], 0\n" \
  "s_nop 0\n" \
  "buffer_load_dwordx4 %[A3], %[wrs], s67 offen offset:0 lds\n" \
  "buffer_load_dwordx4 %[A3], %[wrs], s68 offen offset:1024 lds\n" \
  "buffer_load_dwordx4 %[A3], %[wrs], s69 offen offset:2048 lds\n" \
  "buffer_load_dwordx4 %[A3], %[wrs], s70 offen offset:3072 lds\n" \
  "v_mfma_f32_16x16x32_bf16 v[32:35], a[144:147], v[52:55], v[32:35]\n" \
  "v_mfma_f32_16x16x32_bf16 v[36:39], a[148:151], v[52:55], v[36:39]\n" \
  "v_mfma_f32_16x16x32_bf16 v[40:43], a[152:155], v[52:55], v[40:43]\n" \
  "v_mfma_f32_16x16x32_bf16 v[44:47], a[156:159], v[52:55], v[44:47]\n" \
  "s_waitcnt vmcnt(8)\n" \
  "ds_read_b64_tr_b16 a[176:177], %[vT0] offset:32768\n" \
  "ds_read_b64_tr_b16 a[178:179], %[vT0] offset:33792\n" \
  "ds_read_b64_tr_b16 a[180:181], %[vT1] offset:32768\n" \
  "ds_read_b64_tr_b16 a[182:183], %[vT1] offset:33792\n" \
  "ds_read_b64_tr_b16 a[184:185], %[vT2] offset:32768\n" \
  "ds_read_b64_tr_b16 a[186:187], %[vT2] offset:33792\n" \
  "ds_read_b64_tr_b16 a[188:189], %[vT3] offset:32768\n" \
  "ds_read_b64_tr_b16 a[190:191], %[vT3] offset:33792\n" \
  "s_waitcnt lgkmcnt(8)\n" \
  "s_lshl_b32 s74, s87, 12\n" \
  "s_add_u32 s67, s74, 262144\n" \
  "s_add_u32 s68, s74, 293888\n" \
  "s_add_u32 s69, s74, 391168\n" \
  "s_add_u32 s70, s74, 422912\n" \
  "s_add_u32 m0, %[w4k], 16384\n" \
  "s_nop 0\n" \
  "buffer_load_dwordx4 %[A3], %[wrs], s67 offen offset:0 lds\n" \
  "buffer_load_dwordx4 %[A3], %[wrs], s68 offen offset:1024 lds\n" \
  "buffer_load_dwordx4 %[A3], %[wrs], s69 offen offset:2048 lds\n" \
  "buffer_load_dwordx4 %[A3], %[wrs], s70 offen offset:3072 lds\n" \
  "v_mfma_f32_16x16x32_bf16 v[32:35], a[160:163], v[56:59], v[32:35]\n" \
  "v_mfma_f32_16x16x32_bf16 v[36:39], a[164:167], v[56:59], v[36:39]\n" \
  "v_mfma_f32_16x16x32_bf16 v[40:43], a[168:171], v[56:59], v[40:43]\n" \
  "v_mfma_f32_16x16x32_bf16 v[44:47], a[172:175], v[56:59], v[44:47]\n" \
  "s_waitcnt vmcnt(8)\n" \
  "s_waitcnt lgkmcnt(0)\n" \
  "v_mfma_f32_16x16x32_bf16 v[32:35], a[176:179], v[60:63], v[32:35]\n" \
  "v_mfma_f32_16x16x32_bf16 v[36:39], a[180:183], v[60:63], v[36:39]\n" \
  "v_mfma_f32_16x16x32_bf16 v[40:43], a[184:187], v[60:63], v[40:43]\n" \
  "v_mfma_f32_16x16x32_bf16 v[44:47], a[188:191], v[60:63], v[44:47]\n" \
  "ds_read_b128 v[48:51], %[vX0] offset:49152\n" \
  "ds_read_b128 v[52:55], %[vX1] offset:49152\n" \
  "ds_read_b128 v[56:59], %[vX0] offset:51200\n" \
  "ds_read_b128 v[60:63], %[vX1] offset:51200\n" \
  "s_lshl_b32 s74, s87, 12\n" \
  "s_add_u32 s67, s74, 524288\n" \
  "s_add_u32 s68, s74, 556032\n" \
  "s_add_u32 s69, s74, 653312\n" \
  "s_add_u32 s70, s74, 685056\n" \
  "s_add_u32 m0, %[w4k], 32768\n" \
  "s_nop 0\n" \
  "buffer_load_dwordx4 %[A3], %[wrs], s67 offen offset:0 lds\n" \
  "buffer_load_dwordx4 %[A3], %[wrs], s68 offen offset:1024 lds\n" \
  "buffer_load_dwordx4 %[A3], %[wrs], s69 offen offset:2048 lds\n" \
  "buffer_load_dwordx4 %[A3], %[wrs], s70 offen offset:3072 lds\n" \
  "s_waitcnt vmcnt(8)\n" \
  "ds_read_b64_tr_b16 a[192:193], %[vT0] offset:0\n" \
  "ds_read_b64_tr_b16 a[194:195], %[vT0] offset:1024\n" \
  "ds_read_b64_tr_b16 a[196:197], %[vT1] offset:0\n" \
  "ds_read_b64_tr_b16 a[198:199], %[vT1] offset:1024\n" \
  "ds_read_b64_tr_b16 a[200:201], %[vT2] offset:0\n" \
  "ds_read_b64_tr_b16 a[202:203], %[vT2] offset:1024\n" \
  "ds_read_b64_tr_b16 a[204:205], %[vT3] offset:0\n" \
  "ds_read_b64_tr_b16 a[206:207], %[vT3] offset:1024\n" \
  "s_waitcnt lgkmcnt(8)\n" \
  "s_lshl_b32 s74, s87, 12\n" \
  "s_add_u32 s67, s74, 786432\n" \
  "s_add_u32 s68, s74, 818176\n" \
  "s_add_u32 s69, s74, 915456\n" \
  "s_add_u32 s70, s74, 947200\n" \
  "s_add_u32 m0, %[w4k], 49152\n" \
  "s_nop 0\n" \
  "buffer_load_dwordx4 %[A3], %[wrs], s67 offen offset:0 lds\n" \
  "buffer_load_dwordx4 %[A3], %[wrs], s68 offen offset:1024 lds\n" \
  "buffer_load_dwordx4 %[A3], %[wrs], s69 offen offset:2048 lds\n" \
  "buffer_load_dwordx4 %[A3], %[wrs], s70 offen offset:3072 lds\n" \
  "s_waitcnt vmcnt(8)\n" \
  "ds_read_b64_tr_b16 a[208:209], %[vT0] offset:16384\n" \
  "ds_read_b64_tr_b16 a[210:211], %[vT0] offset:17408\n" \
  "ds_read_b64_tr_b16 a[212:213], %[vT1] offset:16384\n" \
  "ds_read_b64_tr_b16 a[214:215], %[vT1] offset:17408\n" \
  "ds_read_b64_tr_b16 a[216:217], %[vT2] offset:16384\n" \
  "ds_read_b64_tr_b16 a[218:219], %[vT2] offset:17408\n" \
  "ds_read_b64_tr_b16 a[220:221], %[vT3] offset:16384\n" \
  "ds_read_b64_tr_b16 a[222:223], %[vT3] offset:17408\n" \
  "s_waitcnt lgkmcnt(8)\n" \
  "s_add_u32 s65, %[xrow], s88\n" \
  "s_add_u32 s66, s65, 64512\n" \
  "s_add_u32 m0, %[w4k], 0\n" \
  "s_nop 0\n" \
  "buffer_load_dwordx4 %[A1], %[xrs], s65 offen lds\n" \
  "buffer_load_dwordx4 %[A1], %[xrs], s66 offen offset:1024 lds\n" \
  "buffer_load_dwordx4 %[A2], %[xrs], s65 offen offset:2048 lds\n" \
  "buffer_load_dwordx4 %[A2], %[xrs], s66 offen offset:3072 lds\n" \
  "v_mfma_f32_16x16x32_bf16 v[32:35], a[192:195], v[48:51], v[32:35]\n" \
  "v_mfma_f32_16x16x32_bf16 v[36:39], a[196:199], v[48:51], v[36:39]\n" \
  "v_mfma_f32_16x16x32_bf16 v[40:43], a[200:203], v[48:51], v[40:43]\n" \
  "v_mfma_f32_16x16x32_bf16 v[44:47], a[204:207], v[48:51], v[44:47]\n" \
  "s_waitcnt vmcnt(8)\n" \
  "ds_read_b64_tr_b16 a[224:225], %[vT0] offset:32768\n" \
  "ds_read_b64_tr_b16 a[226:227], %[vT0] offset:33792\n" \
  "ds_read_b64_tr_b16 a[228:229], %[vT1] offset:32768\n" \
  "ds_read_b64_tr_b16 a[230:231], %[vT1] offset:33792\n" \
  "ds_read_b64_tr_b16 a[232:233], %[vT2] offset:32768\n" \
  "ds_read_b64_tr_b16 a[234:235], %[vT2] offset:33792\n" \
  "ds_read_b64_tr_b16 a[236:237], %[vT3] offset:32768\n" \
  "ds_read_b64_tr_b16 a[238:239], %[vT3] offset:33792\n" \
  "s_waitcnt lgkmcnt(8)\n" \
  "s_lshl_b32 s74, s88, 12\n" \
  "s_add_u32 s67, s74, 0\n" \
  "s_add_u32 s68, s74, 31744\n" \
  "s_add_u32 s69, s74, 129024\n" \
  "s_add_u32 s70, s74, 160768\n" \
  "s_add_u32 m0, %[w4k], 16384\n" \
  "s_nop 0\n" \
  "buffer_load_dwordx4 %[A3], %[wrs], s67 offen offset:0 lds\n" \
  "buffer_load_dwordx4 %[A3], %[wrs], s68 offen offset:1024 lds\n" \
  "buffer_load_dwordx4 %[A3], %[wrs], s69 offen offset:2048 lds\n" \
  "buffer_load_dwordx4 %[A3], %[wrs], s70 offen offset:3072 lds\n" \
  "v_mfma_f32_16x16x32_bf16 v[32:35], a[208:211], v[52:55], v[32:35]\n" \
  "v_mfma_f32_16x16x32_bf16 v[36:39], a[212:215], v[52:55], v[36:39]\n" \
  "v_mfma_f32_16x16x32_bf16 v[40:43], a[216:219], v[52:55], v[40:43]\n" \
  "v_mfma_f32_16x16x32_bf16 v[44:47], a[220:223], v[52:55], v[44:47]\n" \
  "s_waitcnt vmcnt(8)\n" \
  "ds_read_b64_tr_b16 a[240:241], %[vT0] offset:49152\n" \
  "ds_read_b64_tr_b16 a[242:243], %[vT0] offset:50176\n" \
  "ds_read_b64_tr_b16 a[244:245], %[vT1] offset:49152\n" \
  "ds_read_b64_tr_b16 a[246:247], %[vT1] offset:50176\n" \
  "ds_read_b64_tr_b16 a[248:249], %[vT2] offset:49152\n" \
  "ds_read_b64_tr_b16 a[250:251], %[vT2] offset:50176\n" \
  "ds_read_b64_tr_b16 a[252:253], %[vT3] offset:49152\n" \
  "ds_read_b64_tr_b16 a[254:255], %[vT3] offset:50176\n" \
  "s_waitcnt lgkmcnt(8)\n" \
  "s_lshl_b32 s74, s88, 12\n" \
  "s_add_u32 s67, s74, 262144\n" \
  "s_add_u32 s68, s74, 293888\n" \
  "s_add_u32 s69, s74, 391168\n" \
  "s_add_u32 s70, s74, 422912\n" \
  "s_add_u32 m0, %[w4k], 32768\n" \
  "s_nop 0\n" \
  "buffer_load_dwordx4 %[A3], %[wrs], s67 offen offset:0 lds\n" \
  "buffer_load_dwordx4 %[A3], %[wrs], s68 offen offset:1024 lds\n" \
  "buffer_load_dwordx4 %[A3], %[wrs], s69 offen offset:2048 lds\n" \
  "buffer_load_dwordx4 %[A3], %[wrs], s70 offen offset:3072 lds\n" \
  "v_mfma_f32_16x16x32_bf16 v[32:35], a[224:227], v[56:59], v[32:35]\n" \
  "v_mfma_f32_16x16x32_bf16 v[36:39], a[228:231], v[56:59], v[36:39]\n" \
  "v_mfma_f32_16x16x32_bf16 v[40:43], a[232:235], v[56:59], v[40:43]\n" \
  "v_mfma_f32_16x16x32_bf16 v[44:47], a[236:239], v[56:59], v[44:47]\n" \
  "s_waitcnt vmcnt(8)\n" \
  "s_waitcnt lgkmcnt(0)\n" \
  "v_mfma_f32_16x16x32_bf16 v[32:35], a[240:243], v[60:63], v[32:35]\n" \
  "v_mfma_f32_16x16x32_bf16 v[36:39], a[244:247], v[60:63], v[36:39]\n" \
  "v_mfma_f32_16x16x32_bf16 v[40:43], a[248:251], v[60:63], v[40:43]\n" \
  "v_mfma_f32_16x16x32_bf16 v[44:47], a[252:255], v[60:63], v[44:47]\n" \
  "ds_read_b128 v[48:51], %[vX0] offset:0\n" \
  "ds_read_b128 v[52:55], %[vX1] offset:0\n" \
  "ds_read_b128 v[56:59], %[vX0] offset:2048\n" \
  "ds_read_b128 v[60:63], %[vX1] offset:2048\n" \
  "s_lshl_b32 s74, s88, 12\n" \
  "s_add_u32 s67, s74, 524288\n" \
  "s_add_u32 s68, s74, 556032\n" \
  "s_add_u32 s69, s74, 653312\n" \
  "s_add_u32 s70, s74, 685056\n" \
  "s_add_u32 m0, %[w4k], 65536\n" \
  "s_nop 0\n" \
  "buffer_load_dwordx4 %[A3], %[wrs], s67 offen offset:0 lds\n" \
  "buffer_load_dwordx4 %[A3], %[wrs], s68 offen offset:1024 lds\n" \
  "buffer_load_dwordx4 %[A3], %[wrs], s69 offen offset:2048 lds\n" \
  "buffer_load_dwordx4 %[A3], %[wrs], s70 offen offset:3072 lds\n" \
  "s_waitcnt vmcnt(8)\n" \
  "ds_read_b64_tr_b16 v[80:81], %[vT0] offset:16384\n" \
  "ds_read_b64_tr_b16 v[82:83], %[vT0] offset:17408\n" \
  "ds_read_b64_tr_b16 v[84:85], %[vT1] offset:16384\n" \
  "ds_read_b64_tr_b16 v[86:87], %[vT1] offset:17408\n" \
  "ds_read_b64_tr_b16 v[88:89], %[vT2] offset:16384\n" \
  "ds_read_b64_tr_b16 v[90:91], %[vT2] offset:17408\n" \
  "ds_read_b64_tr_b16 v[92:93], %[vT3] offset:16384\n" \
  "ds_read_b64_tr_b16 v[94:95], %[vT3] offset:17408\n" \
  "s_waitcnt lgkmcnt(8)\n" \
  "s_lshl_b32 s74, s88, 12\n" \
  "s_add_u32 s67, s74, 786432\n" \
  "s_add_u32 s68, s74, 818176\n" \
  "s_add_u32 s69, s74, 915456\n" \
  "s_add_u32 s70, s74, 947200\n" \
  "s_add_u32 m0, %[w4k], 0\n" \
  "s_nop 0\n" \
  "buffer_load_dwordx4 %[A3], %[wrs], s67 offen offset:0 lds\n" \
  "buffer_load_dwordx4 %[A3], %[wrs], s68 offen offset:1024 lds\n" \
  "buffer_load_dwordx4 %[A3], %[wrs], s69 offen offset:2048 lds\n" \
  "buffer_load_dwordx4 %[A3], %[wrs], s70 offen offset:3072 lds\n" \
  "s_waitcnt vmcnt(8)\n" \
  "ds_read_b64_tr_b16 v[96:97], %[vT0] offset:32768\n" \
  "ds_read_b64_tr_b16 v[98:99], %[vT0] offset:33792\n" \
  "ds_read_b64_tr_b16 v[100:101], %[vT1] offset:32768\n" \
  "ds_read_b64_tr_b16 v[102:103], %[vT1] offset:33792\n" \
  "ds_read_b64_tr_b16 v[104:105], %[vT2] offset:32768\n" \
  "ds_read_b64_tr_b16 v[106:107], %[vT2] offset:33792\n" \
  "ds_read_b64_tr_b16 v[108:109], %[vT3] offset:32768\n" \
  "ds_read_b64_tr_b16 v[110:111], %[vT3] offset:33792\n" \
  "s_waitcnt lgkmcnt(8)\n" \
  "s_add_u32 s65, %[xrow], s89\n" \
  "s_add_u32 s66, s65, 64512\n" \
  "s_add_u32 m0, %[w4k], 16384\n" \
  "s_nop 0\n" \
  "buffer_load_dwordx4 %[A1], %[xrs], s65 offen lds\n" \
  "buffer_load_dwordx4 %[A1], %[xrs], s66 offen offset:1024 lds\n" \
  "buffer_load_dwordx4 %[A2], %[xrs], s65 offen offset:2048 lds\n" \
  "buffer_load_dwordx4 %[A2], %[xrs], s66 offen offset:3072 lds\n" \
  "v_mfma_f32_16x16x32_bf16 v[32:35], v[80:83], v[48:51], v[32:35]\n" \
  "v_mfma_f32_16x16x32_bf16 v[36:39], v[84:87], v[48:51], v[36:39]\n" \
  "v_mfma_f32_16x16x32_bf16 v[40:43], v[88:91], v[48:51], v[40:43]\n" \
  "v_mfma_f32_16x16x32_bf16 v[44:47], v[92:95], v[48:51], v[44:47]\n" \
  "s_waitcnt vmcnt(8)\n" \
  "s_waitcnt lgkmcnt(0)\n" \
  "v_mfma_f32_16x16x32_bf16 v[32:35], v[96:99], v[52:55], v[32:35]\n" \
  "v_mfma_f32_16x16x32_bf16 v[36:39], v[100:103], v[52:55], v[36:39]\n" \
  "v_mfma_f32_16x16x32_bf16 v[40:43], v[104:107], v[52:55], v[40:43]\n" \
  "v_mfma_f32_16x16x32_bf16 v[44:47], v[108:111], v[52:55], v[44:47]\n" \
  "v_add_u32 v20, 65536, %[vT0]\n" \
  "v_add_u32 v21, 65536, %[vT1]\n" \
  "v_add_u32 v22, 65536, %[vT2]\n" \
  "v_add_u32 v23, 65536, %[vT3]\n" \
  "ds_read_b64_tr_b16 v[64:65], v20 offset:0\n" \
  "ds_read_b64_tr_b16 v[66:67], v20 offset:1024\n" \
  "ds_read_b64_tr_b16 v[68:69], v21 offset:0\n" \
  "ds_read_b64_tr_b16 v[70:71], v21 offset:1024\n" \
  "ds_read_b64_tr_b16 v[72:73], v22 offset:0\n" \
  "ds_read_b64_tr_b16 v[74:75], v22 offset:1024\n" \
  "ds_read_b64_tr_b16 v[76:77], v23 offset:0\n" \
  "ds_read_b64_tr_b16 v[78:79], v23 offset:1024\n" \
  "s_lshl_b32 s74, s89, 12\n" \
  "s_add_u32 s67, s74, 0\n" \
  "s_add_u32 s68, s74, 31744\n" \
  "s_add_u32 s69, s74, 129024\n" \
  "s_add_u32 s70, s74, 160768\n" \
  "s_add_u32 m0, %[w4k], 32768\n" \
  "s_nop 0\n" \
  "buffer_load_dwordx4 %[A3], %[wrs], s67 offen offset:0 lds\n" \
  "buffer_load_dwordx4 %[A3], %[wrs], s68 offen offset:1024 lds\n" \
  "buffer_load_dwordx4 %[A3], %[wrs], s69 offen offset:2048 lds\n" \
  "buffer_load_dwordx4 %[A3], %[wrs], s70 offen offset:3072 lds\n" \
  "s_waitcnt vmcnt(8)\n" \
  "ds_read_b64_tr_b16 v[112:113], %[vT0] offset:0\n" \
  "ds_read_b64_tr_b16 v[114:115], %[vT0] offset:1024\n" \
  "ds_read_b64_tr_b16 v[116:117], %[vT1] offset:0\n" \
  "ds_read_b64_tr_b16 v[118:119], %[vT1] offset:1024\n" \
  "ds_read_b64_tr_b16 v[120:121], %[vT2] offset:0\n" \
  "ds_read_b64_tr_b16 v[122:123], %[vT2] offset:1024\n" \
  "ds_read_b64_tr_b16 v[124:125], %[vT3] offset:0\n" \
  "ds_read_b64_tr_b16 v[126:127], %[vT3] offset:1024\n" \
  "s_waitcnt lgkmcnt(8)\n" \
  "s_lshl_b32 s74, s89, 12\n" \
  "s_add_u32 s67, s74, 262144\n" \
  "s_add_u32 s68, s74, 293888\n" \
  "s_add_u32 s69, s74, 391168\n" \
  "s_add_u32 s70, s74, 422912\n" \
  "s_add_u32 m0, %[w4k], 81920\n" \
  "s_nop 0\n" \
  "buffer_load_dwordx4 %[A3], %[wrs], s67 offen offset:0 lds\n" \
  "buffer_load_dwordx4 %[A3], %[wrs], s68 offen offset:1024 lds\n" \
  "buffer_load_dwordx4 %[A3], %[wrs], s69 offen offset:2048 lds\n" \
  "buffer_load_dwordx4 %[A3], %[wrs], s70 offen offset:3072 lds\n" \
  "v_mfma_f32_16x16x32_bf16 v[32:35], v[64:67], v[56:59], v[32:35]\n" \
  "v_mfma_f32_16x16x32_bf16 v[36:39], v[68:71], v[56:59], v[36:39]\n" \
  "v_mfma_f32_16x16x32_bf16 v[40:43], v[72:75], v[56:59], v[40:43]\n" \
  "v_mfma_f32_16x16x32_bf16 v[44:47], v[76:79], v[56:59], v[44:47]\n" \
  "ds_write_b128 %[vL16], v[64:67] offset:0\n" \
  "ds_write_b128 %[vL16], v[68:71] offset:1024\n" \
  "ds_write_b128 %[vL16], v[72:75] offset:2048\n" \
  "ds_write_b128 %[vL16], v[76:79] offset:3072\n" \
  "s_waitcnt vmcnt(8)\n" \
  "s_waitcnt lgkmcnt(4)\n" \
  "v_mfma_f32_16x16x32_bf16 v[32:35], v[112:115], v[60:63], v[32:35]\n" \
  "v_mfma_f32_16x16x32_bf16 v[36:39], v[116:119], v[60:63], v[36:39]\n" \
  "v_mfma_f32_16x16x32_bf16 v[40:43], v[120:123], v[60:63], v[40:43]\n" \
  "v_mfma_f32_16x16x32_bf16 v[44:47], v[124:127], v[60:63], v[44:47]\n" \
  "ds_read_b128 v[48:51], %[vX0] offset:16384\n" \
  "ds_read_b128 v[52:55], %[vX1] offset:16384\n" \
  "ds_read_b128 v[56:59], %[vX0] offset:18432\n" \
  "ds_read_b128 v[60:63], %[vX1] offset:18432\n" \
  "s_lshl_b32 s74, s89, 12\n" \
  "s_add_u32 s67, s74, 524288\n" \
  "s_add_u32 s68, s74, 556032\n" \
  "s_add_u32 s69, s74, 653312\n" \
  "s_add_u32 s70, s74, 685056\n" \
  "s_add_u32 m0, %[w4k], 0\n" \
  "s_nop 0\n" \
  "buffer_load_dwordx4 %[A3], %[wrs], s67 offen offset:0 lds\n" \
  "buffer_load_dwordx4 %[A3], %[wrs], s68 offen offset:1024 lds\n" \
  "buffer_load_dwordx4 %[A3], %[wrs], s69 offen offset:2048 lds\n" \
  "buffer_load_dwordx4 %[A3], %[wrs], s70 offen offset:3072 lds\n" \
  "s_waitcnt vmcnt(8)\n" \
  "ds_read_b64_tr_b16 v[128:129], %[vT0] offset:32768\n" \
  "ds_read_b64_tr_b16 v[130:131], %[vT0] offset:33792\n" \
  "ds_read_b64_tr_b16 v[132:133], %[vT1] offset:32768\n" \
  "ds_read_b64_tr_b16 v[134:135], %[vT1] offset:33792\n" \
  "ds_read_b64_tr_b16 v[136:137], %[vT2] offset:32768\n" \
  "ds_read_b64_tr_b16 v[138:139], %[vT2] offset:33792\n" \
  "ds_read_b64_tr_b16 v[140:141], %[vT3] offset:32768\n" \
  "ds_read_b64_tr_b16 v[142:143], %[vT3] offset:33792\n" \
  "s_waitcnt lgkmcnt(8)\n" \
  "s_lshl_b32 s74, s89, 12\n" \
  "s_add_u32 s67, s74, 786432\n" \
  "s_add_u32 s68, s74, 818176\n" \
  "s_add_u32 s69, s74, 915456\n" \
  "s_add_u32 s70, s74, 947200\n" \
  "s_add_u32 m0, %[w4k], 16384\n" \
  "s_nop 0\n" \
  "buffer_load_dwordx4 %[A3], %[wrs], s67 offen offset:0 lds\n" \
  "buffer_load_dwordx4 %[A3], %[wrs], s68 offen offset:1024 lds\n" \
  "buffer_load_dwordx4 %[A3], %[wrs], s69 offen offset:2048 lds\n" \
  "buffer_load_dwordx4 %[A3], %[wrs], s70 offen offset:3072 lds\n" \
  "s_waitcnt vmcnt(8)\n" \
  "s_waitcnt lgkmcnt(0)\n" \
  "v_mfma_f32_16x16x32_bf16 v[32:35], v[128:131], v[48:51], v[32:35]\n" \
  "v_mfma_f32_16x16x32_bf16 v[36:39], v[132:135], v[48:51], v[36:39]\n" \
  "v_mfma_f32_16x16x32_bf16 v[40:43], v[136:139], v[48:51], v[40:43]\n" \
  "v_mfma_f32_16x16x32_bf16 v[44:47], v[140:143], v[48:51], v[44:47]\n" \
  "v_add_u32 v20, 65536, %[vT0]\n" \
  "v_add_u32 v21, 65536, %[vT1]\n" \
  "v_add_u32 v22, 65536, %[vT2]\n" \
  "v_add_u32 v23, 65536, %[vT3]\n" \
  "ds_read_b64_tr_b16 v[64:65], v20 offset:16384\n" \
  "ds_read_b64_tr_b16 v[66:67], v20 offset:17408\n" \
  "ds_read_b64_tr_b16 v[68:69], v21 offset:16384\n" \
  "ds_read_b64_tr_b16 v[70:71], v21 offset:17408\n" \
  "ds_read_b64_tr_b16 v[72:73], v22 offset:16384\n" \
  "ds_read_b64_tr_b16 v[74:75], v22 offset:17408\n" \
  "ds_read_b64_tr_b16 v[76:77], v23 offset:16384\n" \
  "ds_read_b64_tr_b16 v[78:79], v23 offset:17408\n" \
  "s_add_u32 s65, %[xrow], s90\n" \
  "s_add_u32 s66, s65, 64512\n" \
  "s_add_u32 m0, %[w4k], 32768\n" \
  "s_nop 0\n" \
  "buffer_load_dwordx4 %[A1], %[xrs], s65 offen lds\n" \
  "buffer_load_dwordx4 %[A1], %[xrs], s66 offen offset:1024 lds\n" \
  "buffer_load_dwordx4 %[A2], %[xrs], s65 offen offset:2048 lds\n" \
  "buffer_load_dwordx4 %[A2], %[xrs], s66 offen offset:3072 lds\n" \
  "s_waitcnt vmcnt(8)\n" \
  "ds_read_b64_tr_b16 v[144:145], %[vT0] offset:0\n" \
  "ds_read_b64_tr_b16 v[146:147], %[vT0] offset:1024\n" \
  "ds_read_b64_tr_b16 v[148:149], %[vT1] offset:0\n" \
  "ds_read_b64_tr_b16 v[150:151], %[vT1] offset:1024\n" \
  "ds_read_b64_tr_b16 v[152:153], %[vT2] offset:0\n" \
  "ds_read_b64_tr_b16 v[154:155], %[vT2] offset:1024\n" \
  "ds_read_b64_tr_b16 v[156:157], %[vT3] offset:0\n" \
  "ds_read_b64_tr_b16 v[158:159], %[vT3] offset:1024\n" \
  "s_waitcnt lgkmcnt(8)\n" \
  "s_lshl_b32 s74, s90, 12\n" \
  "s_add_u32 s67, s74, 0\n" \
  "s_add_u32 s68, s74, 31744\n" \
  "s_add_u32 s69, s74, 129024\n" \
  "s_add_u32 s70, s74, 160768\n" \
  "s_add_u32 m0, %[w4k], 98304\n" \
  "s_nop 0\n" \
  "buffer_load_dwordx4 %[A3], %[wrs], s67 offen offset:0 lds\n" \
  "buffer_load_dwordx4 %[A3], %[wrs], s68 offen offset:1024 lds\n" \
  "buffer_load_dwordx4 %[A3], %[wrs], s69 offen offset:2048 lds\n" \
  "buffer_load_dwordx4 %[A3], %[wrs], s70 offen offset:3072 lds\n" \
  "v_mfma_f32_16x16x32_bf16 v[32:35], v[64:67], v[52:55], v[32:35]\n" \
  "v_mfma_f32_16x16x32_bf16 v[36:39], v[68:71], v[52:55], v[36:39]\n" \
  "v_mfma_f32_16x16x32_bf16 v[40:43], v[72:75], v[52:55], v[40:43]\n" \
  "v_mfma_f32_16x16x32_bf16 v[44:47], v[76:79], v[52:55], v[44:47]\n" \
  "ds_write_b128 %[vL16], v[64:67] offset:16384\n" \
  "ds_write_b128 %[vL16], v[68:71] offset:17408\n" \
  "ds_write_b128 %[vL16], v[72:75] offset:18432\n" \
  "ds_write_b128 %[vL16], v[76:79] offset:19456\n" \
  "s_waitcnt vmcnt(8)\n" \
  "ds_read_b64_tr_b16 v[160:161], %[vT0] offset:16384\n" \
  "ds_read_b64_tr_b16 v[162:163], %[vT0] offset:17408\n" \
  "ds_read_b64_tr_b16 v[164:165], %[vT1] offset:16384\n" \
  "ds_read_b64_tr_b16 v[166:167], %[vT1] offset:17408\n" \
  "ds_read_b64_tr_b16 v[168:169], %[vT2] offset:16384\n" \
  "ds_read_b64_tr_b16 v[170:171], %[vT2] offset:17408\n" \
  "ds_read_b64_tr_b16 v[172:173], %[vT3] offset:16384\n" \
  "ds_read_b64_tr_b16 v[174:175], %[vT3] offset:17408\n" \
  "s_waitcnt lgkmcnt(12)\n" \
  "s_lshl_b32 s74, s90, 12\n" \
  "s_add_u32 s67, s74, 262144\n" \
  "s_add_u32 s68, s74, 293888\n" \
  "s_add_u32 s69, s74, 391168\n" \
  "s_add_u32 s70, s74, 422912\n" \
  "s_add_u32 m0, %[w4k], 0\n" \
  "s_nop 0\n" \
  "buffer_load_dwordx4 %[A3], %[wrs], s67 offen offset:0 lds\n" \
  "buffer_load_dwordx4 %[A3], %[wrs], s68 offen offset:1024 lds\n" \
  "buffer_load_dwordx4 %[A3], %[wrs], s69 offen offset:2048 lds\n" \
  "buffer_load_dwordx4 %[A3], %[wrs], s70 offen offset:3072 lds\n" \
  "v_mfma_f32_16x16x32_bf16 v[32:35], v[144:147], v[56:59], v[32:35]\n" \
  "v_mfma_f32_16x16x32_bf16 v[36:39], v[148:151], v[56:59], v[36:39]\n" \
  "v_mfma_f32_16x16x32_bf16 v[40:43], v[152:155], v[56:59], v[40:43]\n" \
  "v_mfma_f32_16x16x32_bf16 v[44:47], v[156:159], v[56:59], v[44:47]\n" \
  "s_waitcnt vmcnt(8)\n" \
  "s_waitcnt lgkmcnt(0)\n" \
  "v_mfma_f32_16x16x32_bf16 v[32:35], v[160:163], v[60:63], v[32:35]\n" \
  "v_mfma_f32_16x16x32_bf16 v[36:39], v[164:167], v[60:63], v[36:39]\n" \
  "v_mfma_f32_16x16x32_bf16 v[40:43], v[168:171], v[60:63], v[40:43]\n" \
  "v_mfma_f32_16x16x32_bf16 v[44:47], v[172:175], v[60:63], v[44:47]\n" \
  "ds_read_b128 v[48:51], %[vX0] offset:32768\n" \
  "ds_read_b128 v[52:55], %[vX1] offset:32768\n" \
  "ds_read_b128 v[56:59], %[vX0] offset:34816\n" \
  "ds_read_b128 v[60:63], %[vX1] offset:34816\n" \
  "s_lshl_b32 s74, s90, 12\n" \
  "s_add_u32 s67, s74, 524288\n" \
  "s_add_u32 s68, s74, 556032\n" \
  "s_add_u32 s69, s74, 653312\n" \
  "s_add_u32 s70, s74, 685056\n" \
  "s_add_u32 m0, %[w4k], 16384\n" \
  "s_nop 0\n" \
  "buffer_load_dwordx4 %[A3], %[wrs], s67 offen offset:0 lds\n" \
  "buffer_load_dwordx4 %[A3], %[wrs], s68 offen offset:1024 lds\n" \
  "buffer_load_dwordx4 %[A3], %[wrs], s69 offen offset:2048 lds\n" \
  "buffer_load_dwordx4 %[A3], %[wrs], s70 offen offset:3072 lds\n" \
  "s_waitcnt vmcnt(8)\n" \
  "v_add_u32 v20, 65536, %[vT0]\n" \
  "v_add_u32 v21, 65536, %[vT1]\n" \
  "v_add_u32 v22, 65536, %[vT2]\n" \
  "v_add_u32 v23, 65536, %[vT3]\n" \
  "ds_read_b64_tr_b16 v[64:65], v20 offset:32768\n" \
  "ds_read_b64_tr_b16 v[66:67], v20 offset:33792\n" \
  "ds_read_b64_tr_b16 v[68:69], v21 offset:32768\n" \
  "ds_read_b64_tr_b16 v[70:71], v21 offset:33792\n" \
  "ds_read_b64_tr_b16 v[72:73], v22 offset:32768\n" \
  "ds_read_b64_tr_b16 v[74:75], v22 offset:33792\n" \
  "ds_read_b64_tr_b16 v[76:77], v23 offset:32768\n" \
  "ds_read_b64_tr_b16 v[78:79], v23 offset:33792\n" \
  "s_waitcnt lgkmcnt(8)\n" \
  "s_lshl_b32 s74, s90, 12\n" \
  "s_add_u32 s67, s74, 786432\n" \
  "s_add_u32 s68, s74, 818176\n" \
  "s_add_u32 s69, s74, 915456\n" \
  "s_add_u32 s70, s74, 947200\n" \
  "s_add_u32 m0, %[w4k], 114688\n" \
  "s_nop 0\n" \
  "buffer_load_dwordx4 %[A3], %[wrs], s67 offen offset:0 lds\n" \
  "buffer_load_dwordx4 %[A3], %[wrs], s68 offen offset:1024 lds\n" \
  "buffer_load_dwordx4 %[A3], %[wrs], s69 offen offset:2048 lds\n" \
  "buffer_load_dwordx4 %[A3], %[wrs], s70 offen offset:3072 lds\n" \
  "s_waitcnt vmcnt(8)\n" \
  "ds_read_b64_tr_b16 v[176:177], %[vT0] offset:0\n" \
  "ds_read_b64_tr_b16 v[178:179], %[vT0] offset:1024\n" \
  "ds_read_b64_tr_b16 v[180:181], %[vT1] offset:0\n" \
  "ds_read_b64_tr_b16 v[182:183], %[vT1] offset:1024\n" \
  "ds_read_b64_tr_b16 v[184:185], %[vT2] offset:0\n" \
  "ds_read_b64_tr_b16 v[186:187], %[vT2] offset:1024\n" \
  "ds_read_b64_tr_b16 v[188:189], %[vT3] offset:0\n" \
  "ds_read_b64_tr_b16 v[190:191], %[vT3] offset:1024\n" \
  "s_waitcnt lgkmcnt(8)\n" \
  "s_add_u32 s65, %[xrow], s91\n" \
  "s_add_u32 s66, s65, 64512\n" \
  "s_add_u32 m0, %[w4k], 32768\n" \
  "s_nop 0\n" \
  "buffer_load_dwordx4 %[A1], %[xrs], s65 offen lds\n" \
  "buffer_load_dwordx4 %[A1], %[xrs], s66 offen offset:1024 lds\n" \
  "buffer_load_dwordx4 %[A2], %[xrs], s65 offen offset:2048 lds\n" \
  "buffer_load_dwordx4 %[A2], %[xrs], s66 offen offset:3072 lds\n" \
  "v_mfma_f32_16x16x32_bf16 v[32:35], v[64:67], v[48:51], v[32:35]\n" \
  "v_mfma_f32_16x16x32_bf16 v[36:39], v[68:71], v[48:51], v[36:39]\n" \
  "v_mfma_f32_16x16x32_bf16 v[40:43], v[72:75], v[48:51], v[40:43]\n" \
  "v_mfma_f32_16x16x32_bf16 v[44:47], v[76:79], v[48:51], v[44:47]\n" \
  "ds_write_b128 %[vL16], v[64:67] offset:32768\n" \
  "ds_write_b128 %[vL16], v[68:71] offset:33792\n" \
  "ds_write_b128 %[vL16], v[72:75] offset:34816\n" \
  "ds_write_b128 %[vL16], v[76:79] offset:35840\n" \
  "s_waitcnt vmcnt(8)\n" \
  "ds_read_b64_tr_b16 v[192:193], %[vT0] offset:16384\n" \
  "ds_read_b64_tr_b16 v[194:195], %[vT0] offset:17408\n" \
  "ds_read_b64_tr_b16 v[196:197], %[vT1] offset:16384\n" \
  "ds_read_b64_tr_b16 v[198:199], %[vT1] offset:17408\n" \
  "ds_read_b64_tr_b16 v[200:201], %[vT2] offset:16384\n" \
  "ds_read_b64_tr_b16 v[202:203], %[vT2] offset:17408\n" \
  "ds_read_b64_tr_b16 v[204:205], %[vT3] offset:16384\n" \
  "ds_read_b64_tr_b16 v[206:207], %[vT3] offset:17408\n" \
  "s_waitcnt lgkmcnt(12)\n" \
  "s_lshl_b32 s74, s91, 12\n" \
  "s_add_u32 s67, s74, 0\n" \
  "s_add_u32 s68, s74, 31744\n" \
  "s_add_u32 s69, s74, 129024\n" \
  "s_add_u32 s70, s74, 160768\n" \
  "s_add_u32 m0, %[w4k], 0\n" \
  "s_nop 0\n" \
  "buffer_load_dwordx4 %[A3], %[wrs], s67 offen offset:0 lds\n" \
  "buffer_load_dwordx4 %[A3], %[wrs], s68 offen offset:1024 lds\n" \
  "buffer_load_dwordx4 %[A3], %[wrs], s69 offen offset:2048 lds\n" \
  "buffer_load_dwordx4 %[A3], %[wrs], s70 offen offset:3072 lds\n" \
  "v_mfma_f32_16x16x32_bf16 v[32:35], v[176:179], v[52:55], v[32:35]\n" \
  "v_mfma_f32_16x16x32_bf16 v[36:39], v[180:183], v[52:55], v[36:39]\n" \
  "v_mfma_f32_16x16x32_bf16 v[40:43], v[184:187], v[52:55], v[40:43]\n" \
  "v_mfma_f32_16x16x32_bf16 v[44:47], v[188:191], v[52:55], v[44:47]\n" \
  "s_waitcnt vmcnt(8)\n" \
  "s_waitcnt lgkmcnt(0)\n" \
  "v_mfma_f32_16x16x32_bf16 v[32:35], v[192:195], v[56:59], v[32:35]\n" \
  "v_mfma_f32_16x16x32_bf16 v[36:39], v[196:199], v[56:59], v[36:39]\n" \
  "v_mfma_f32_16x16x32_bf16 v[40:43], v[200:203], v[56:59], v[40:43]\n" \
  "v_mfma_f32_16x16x32_bf16 v[44:47], v[204:207], v[56:59], v[44:47]\n" \
  "v_add_u32 v20, 65536, %[vT0]\n" \
  "v_add_u32 v21, 65536, %[vT1]\n" \
  "v_add_u32 v22, 65536, %[vT2]\n" \
  "v_add_u32 v23, 65536, %[vT3]\n" \
  "ds_read_b64_tr_b16 v[64:65], v20 offset:49152\n" \
  "ds_read_b64_tr_b16 v[66:67], v20 offset:50176\n" \
  "ds_read_b64_tr_b16 v[68:69], v21 offset:49152\n" \
  "ds_read_b64_tr_b16 v[70:71], v21 offset:50176\n" \
  "ds_read_b64_tr_b16 v[72:73], v22 offset:49152\n" \
  "ds_read_b64_tr_b16 v[74:75], v22 offset:50176\n" \
  "ds_read_b64_tr_b16 v[76:77], v23 offset:49152\n" \
  "ds_read_b64_tr_b16 v[78:79], v23 offset:50176\n" \
  "s_lshl_b32 s74, s91, 12\n" \
  "s_add_u32 s67, s74, 262144\n" \
  "s_add_u32 s68, s74, 293888\n" \
  "s_add_u32 s69, s74, 391168\n" \
  "s_add_u32 s70, s74, 422912\n" \
  "s_add_u32 m0, %[w4k], 16384\n" \
  "s_nop 0\n" \
  "buffer_load_dwordx4 %[A3], %[wrs], s67 offen offset:0 lds\n" \
  "buffer_load_dwordx4 %[A3], %[wrs], s68 offen offset:1024 lds\n" \
  "buffer_load_dwordx4 %[A3], %[wrs], s69 offen offset:2048 lds\n" \
  "buffer_load_dwordx4 %[A3], %[wrs], s70 offen offset:3072 lds\n" \
  "s_waitcnt vmcnt(8)\n" \
  "s_waitcnt lgkmcnt(0)\n" \
  "v_mfma_f32_16x16x32_bf16 v[32:35], v[64:67], v[60:63], v[32:35]\n" \
  "v_mfma_f32_16x16x32_bf16 v[36:39], v[68:71], v[60:63], v[36:39]\n" \
  "v_mfma_f32_16x16x32_bf16 v[40:43], v[72:75], v[60:63], v[40:43]\n" \
  "v_mfma_f32_16x16x32_bf16 v[44:47], v[76:79], v[60:63], v[44:47]\n" \
  "ds_write_b128 %[vL16], v[64:67] offset:49152\n" \
  "ds_write_b128 %[vL16], v[68:71] offset:50176\n" \
  "ds_write_b128 %[vL16], v[72:75] offset:51200\n" \
  "ds_write_b128 %[vL16], v[76:79] offset:52224\n" \
  "ds_read_b128 v[48:51], %[vX0] offset:32768\n" \
  "ds_read_b128 v[52:55], %[vX1] offset:32768\n" \
  "ds_read_b128 v[56:59], %[vX0] offset:34816\n" \
  "ds_read_b128 v[60:63], %[vX1] offset:34816\n" \
  "s_lshl_b32 s74, s91, 12\n" \
  "s_add_u32 s67, s74, 524288\n" \
  "s_add_u32 s68, s74, 556032\n" \
  "s_add_u32 s69, s74, 653312\n" \
  "s_add_u32 s70, s74, 685056\n" \
  "s_add_u32 m0, %[w4k], 131072\n" \
  "s_nop 0\n" \
  "buffer_load_dwordx4 %[A3], %[wrs], s67 offen offset:0 lds\n" \
  "buffer_load_dwordx4 %[A3], %[wrs], s68 offen offset:1024 lds\n" \
  "buffer_load_dwordx4 %[A3], %[wrs], s69 offen offset:2048 lds\n" \
  "buffer_load_dwordx4 %[A3], %[wrs], s70 offen offset:3072 lds\n" \
  "s_waitcnt vmcnt(8)\n" \
  "ds_read_b64_tr_b16 v[208:209], %[vT0] offset:0\n" \
  "ds_read_b64_tr_b16 v[210:211], %[vT0] offset:1024\n" \
  "ds_read_b64_tr_b16 v[212:213], %[vT1] offset:0\n" \
  "ds_read_b64_tr_b16 v[214:215], %[vT1] offset:1024\n" \
  "ds_read_b64_tr_b16 v[216:217], %[vT2] offset:0\n" \
  "ds_read_b64_tr_b16 v[218:219], %[vT2] offset:1024\n" \
  "ds_read_b64_tr_b16 v[220:221], %[vT3] offset:0\n" \
  "ds_read_b64_tr_b16 v[222:223], %[vT3] offset:1024\n" \
  "s_waitcnt lgkmcnt(8)\n" \
  "s_lshl_b32 s74, s91, 12\n" \
  "s_add_u32 s67, s74, 786432\n" \
  "s_add_u32 s68, s74, 818176\n" \
  "s_add_u32 s69, s74, 915456\n" \
  "s_add_u32 s70, s74, 947200\n" \
  "s_add_u32 m0, %[w4k], 49152\n" \
  "s_nop 0\n" \
  "buffer_load_dwordx4 %[A3], %[wrs], s67 offen offset:0 lds\n" \
  "buffer_load_dwordx4 %[A3], %[wrs], s68 offen offset:1024 lds\n" \
  "buffer_load_dwordx4 %[A3], %[wrs], s69 offen offset:2048 lds\n" \
  "buffer_load_dwordx4 %[A3], %[wrs], s70 offen offset:3072 lds\n" \
  "s_waitcnt vmcnt(8)\n" \
  "ds_read_b64_tr_b16 v[224:225], %[vT0] offset:16384\n" \
  "ds_read_b64_tr_b16 v[226:227], %[vT0] offset:17408\n" \
  "ds_read_b64_tr_b16 v[228:229], %[vT1] offset:16384\n" \
  "ds_read_b64_tr_b16 v[230:231], %[vT1] offset:17408\n" \
  "ds_read_b64_tr_b16 v[232:233], %[vT2] offset:16384\n" \
  "ds_read_b64_tr_b16 v[234:235], %[vT2] offset:17408\n" \
  "ds_read_b64_tr_b16 v[236:237], %[vT3] offset:16384\n" \
  "ds_read_b64_tr_b16 v[238:239], %[vT3] offset:17408\n" \
  "s_waitcnt lgkmcnt(8)\n" \
  "s_add_u32 s65, %[xrow], s84\n" \
  "s_add_u32 s65, s65, 131072\n" \
  "s_add_u32 s66, s65, 64512\n" \
  "s_add_u32 m0, %[w4k], 0\n" \
  "s_nop 0\n" \
  "buffer_load_dwordx4 %[A1], %[xrs], s65 offen lds\n" \
  "buffer_load_dwordx4 %[A1], %[xrs], s66 offen offset:1024 lds\n" \
  "buffer_load_dwordx4 %[A2], %[xrs], s65 offen offset:2048 lds\n" \
  "buffer_load_dwordx4 %[A2], %[xrs], s66 offen offset:3072 lds\n" \
  "v_mfma_f32_16x16x32_bf16 v[32:35], v[208:211], v[48:51], v[32:35]\n" \
  "v_mfma_f32_16x16x32_bf16 v[36:39], v[212:215], v[48:51], v[36:39]\n" \
  "v_mfma_f32_16x16x32_bf16 v[40:43], v[216:219], v[48:51], v[40:43]\n" \
  "v_mfma_f32_16x16x32_bf16 v[44:47], v[220:223], v[48:51], v[44:47]\n" \
  "s_waitcnt vmcnt(8)\n" \
  "s_waitcnt lgkmcnt(0)\n" \
  "v_mfma_f32_16x16x32_bf16 v[32:35], v[224:227], v[52:55], v[32:35]\n" \
  "v_mfma_f32_16x16x32_bf16 v[36:39], v[228:231], v[52:55], v[36:39]\n" \
  "v_mfma_f32_16x16x32_bf16 v[40:43], v[232:235], v[52:55], v[40:43]\n" \
  "v_mfma_f32_16x16x32_bf16 v[44:47], v[236:239], v[52:55], v[44:47]\n" \
  "v_add_u32 v20, 131072, %[vT0]\n" \
  "v_add_u32 v21, 131072, %[vT1]\n" \
  "v_add_u32 v22, 131072, %[vT2]\n" \
  "v_add_u32 v23, 131072, %[vT3]\n" \
  "ds_read_b64_tr_b16 v[64:65], v20 offset:0\n" \
  "ds_read_b64_tr_b16 v[66:67], v20 offset:1024\n" \
  "ds_read_b64_tr_b16 v[68:69], v21 offset:0\n" \
  "ds_read_b64_tr_b16 v[70:71], v21 offset:1024\n" \
  "ds_read_b64_tr_b16 v[72:73], v22 offset:0\n" \
  "ds_read_b64_tr_b16 v[74:75], v22 offset:1024\n" \
  "ds_read_b64_tr_b16 v[76:77], v23 offset:0\n" \
  "ds_read_b64_tr_b16 v[78:79], v23 offset:1024\n" \
  "s_add_u32 s65, %[xrow], s85\n" \
  "s_add_u32 s65, s65, 131072\n" \
  "s_add_u32 s66, s65, 64512\n" \
  "s_add_u32 m0, %[w4k], 16384\n" \
  "s_nop 0\n" \
  "buffer_load_dwordx4 %[A1], %[xrs], s65 offen lds\n" \
  "buffer_load_dwordx4 %[A1], %[xrs], s66 offen offset:1024 lds\n" \
  "buffer_load_dwordx4 %[A2], %[xrs], s65 offen offset:2048 lds\n" \
  "buffer_load_dwordx4 %[A2], %[xrs], s66 offen offset:3072 lds\n" \
  "s_waitcnt vmcnt(8)\n" \
  "ds_read_b64_tr_b16 v[240:241], %[vT0] offset:49152\n" \
  "ds_read_b64_tr_b16 v[242:243], %[vT0] offset:50176\n" \
  "ds_read_b64_tr_b16 v[244:245], %[vT1] offset:49152\n" \
  "ds_read_b64_tr_b16 v[246:247], %[vT1] offset:50176\n" \
  "ds_read_b64_tr_b16 v[248:249], %[vT2] offset:49152\n" \
  "ds_read_b64_tr_b16 v[250:251], %[vT2] offset:50176\n" \
  "ds_read_b64_tr_b16 v[252:253], %[vT3] offset:49152\n" \
  "ds_read_b64_tr_b16 v[254:255], %[vT3] offset:50176\n" \
  "s_waitcnt lgkmcnt(8)\n" \
  "s_add_u32 s65, %[xrow], s86\n" \
  "s_add_u32 s65, s65, 131072\n" \
  "s_add_u32 s66, s65, 64512\n" \
  "s_add_u32 m0, %[w4k], 32768\n" \
  "s_nop 0\n" \
  "buffer_load_dwordx4 %[A1], %[xrs], s65 offen lds\n" \
  "buffer_load_dwordx4 %[A1], %[xrs], s66 offen offset:1024 lds\n" \
  "buffer_load_dwordx4 %[A2], %[xrs], s65 offen offset:2048 lds\n" \
  "buffer_load_dwordx4 %[A2], %[xrs], s66 offen offset:3072 lds\n" \
  "v_mfma_f32_16x16x32_bf16 v[32:35], v[64:67], v[56:59], v[32:35]\n" \
  "v_mfma_f32_16x16x32_bf16 v[36:39], v[68:71], v[56:59], v[36:39]\n" \
  "v_mfma_f32_16x16x32_bf16 v[40:43], v[72:75], v[56:59], v[40:43]\n" \
  "v_mfma_f32_16x16x32_bf16 v[44:47], v[76:79], v[56:59], v[44:47]\n" \
  "ds_write_b128 %[vL16b], v[64:67] offset:0\n" \
  "ds_write_b128 %[vL16b], v[68:71] offset:1024\n" \
  "ds_write_b128 %[vL16b], v[72:75] offset:2048\n" \
  "ds_write_b128 %[vL16b], v[76:79] offset:3072\n" \
  "s_waitcnt vmcnt(8) lgkmcnt(0)\n" \
  "s_waitcnt lgkmcnt(0)\n" \
  "s_and_b32 s93, s93, 0xffff\n" \
  "s_mov_b32 s94, 0x400000\n" \
  "s_mov_b32 s95, 0x20000\n" \
  "v_mfma_f32_16x16x32_bf16 v[32:35], v[240:243], v[60:63], v[32:35]\n" \
  "v_mfma_f32_16x16x32_bf16 v[36:39], v[244:247], v[60:63], v[36:39]\n" \
  "v_mfma_f32_16x16x32_bf16 v[40:43], v[248:251], v[60:63], v[40:43]\n" \
  "v_mfma_f32_16x16x32_bf16 v[44:47], v[252:255], v[60:63], v[44:47]\n" \
  "ds_read_b128 v[48:51], %[vX0] offset:0\n" \
  "ds_read_b128 v[52:55], %[vX1] offset:0\n" \
  "ds_read_b128 v[56:59], %[vX0] offset:2048\n" \
  "ds_read_b128 v[60:63], %[vX1] offset:2048\n" \
  "s_add_u32 s82, %[xrow], 131072\n" \
  "s_add_u32 s65, %[xrow], s87\n" \
  "s_add_u32 s65, s65, 131072\n" \
  "s_add_u32 s66, s65, 64512\n" \
  "s_add_u32 m0, %[w4k], 49152\n" \
  "s_nop 0\n" \
  "buffer_load_dwordx4 %[A1], %[xrs], s65 offen lds\n" \
  "buffer_load_dwordx4 %[A1], %[xrs], s66 offen offset:1024 lds\n" \
  "s_mov_b32 s64, 7\n" \
  "L_loop_%=:\n" \
  "s_nop 7\n" \
  "ds_write_b128 %[vPw0], v[32:35] offset:0\n" \
  "ds_write_b128 %[vPw1], v[36:39] offset:0\n" \
  "ds_write_b128 %[vPw0], v[40:43] offset:128\n" \
  "ds_write_b128 %[vPw1], v[44:47] offset:128\n" \
  "s_waitcnt lgkmcnt(7)\n" \
  "v_mfma_f32_16x16x32_bf16 v[32:35], a[0:3], v[48:51], 0\n" \
  "v_mfma_f32_16x16x32_bf16 v[36:39], a[4:7], v[48:51], 0\n" \
  "buffer_load_dwordx4 %[A2], %[xrs], s65 offen offset:2048 lds\n" \
  "v_mfma_f32_16x16x32_bf16 v[40:43], a[8:11], v[48:51], 0\n" \
  "v_mfma_f32_16x16x32_bf16 v[44:47], a[12:15], v[48:51], 0\n" \
  "s_waitcnt lgkmcnt(6)\n" \
  "v_mfma_f32_16x16x32_bf16 v[32:35], a[16:19], v[52:55], v[32:35]\n" \
  "v_mfma_f32_16x16x32_bf16 v[36:39], a[20:23], v[52:55], v[36:39]\n" \
  "buffer_load_dwordx4 %[A2], %[xrs], s66 offen offset:3072 lds\n" \
  "v_mfma_f32_16x16x32_bf16 v[40:43], a[24:27], v[52:55], v[40:43]\n" \
  "s_add_u32 s65, s82, s88\n" \
  "s_waitcnt lgkmcnt(0)\n" \
  "s_barrier\n" \
  "v_mfma_f32_16x16x32_bf16 v[44:47], a[28:31], v[52:55], v[44:47]\n" \
  "s_waitcnt vmcnt(8)\n" \
  "ds_read_b128 v[48:51], %[vX0] offset:16384\n" \
  "v_mfma_f32_16x16x32_bf16 v[32:35], a[32:35], v[56:59], v[32:35]\n" \
  "ds_read_b128 v[52:55], %[vX1] offset:16384\n" \
  "s_add_u32 s66, s65, 64512\n" \
  "s_add_u32 m0, %[w4k], 0\n" \
  "ds_read_b128 v[64:67], %[vPr] offset:0\n" \
  "v_mfma_f32_16x16x32_bf16 v[36:39], a[36:39], v[56:59], v[36:39]\n" \
  "buffer_load_dwordx4 %[A1], %[xrs], s65 offen lds\n" \
  "ds_read_b128 v[68:71], %[vPr] offset:4096\n" \
  "v_mfma_f32_16x16x32_bf16 v[40:43], a[40:43], v[56:59], v[40:43]\n" \
  "ds_read_b128 v[72:75], %[vPr] offset:8192\n" \
  "v_mfma_f32_16x16x32_bf16 v[44:47], a[44:47], v[56:59], v[44:47]\n" \
  "ds_read_b128 v[56:59], %[vX0] offset:18432\n" \
  "ds_read_b128 v[76:79], %[vPr] offset:12288\n" \
  "v_mfma_f32_16x16x32_bf16 v[32:35], a[48:51], v[60:63], v[32:35]\n" \
  "v_mfma_f32_16x16x32_bf16 v[36:39], a[52:55], v[60:63], v[36:39]\n" \
  "buffer_load_dwordx4 %[A1], %[xrs], s66 offen offset:1024 lds\n" \
  "v_mfma_f32_16x16x32_bf16 v[40:43], a[56:59], v[60:63], v[40:43]\n" \
  "v_mfma_f32_16x16x32_bf16 v[44:47], a[60:63], v[60:63], v[44:47]\n" \
  "ds_read_b128 v[60:63], %[vX1] offset:18432\n" \
  "s_waitcnt lgkmcnt(7)\n" \
  "v_mfma_f32_16x16x32_bf16 v[32:35], a[64:67], v[48:51], v[32:35]\n" \
  "s_waitcnt lgkmcnt(1)\n" \
  "v_add_f32 v64, v64, v68\n" \
  "v_mfma_f32_16x16x32_bf16 v[36:39], a[68:71], v[48:51], v[36:39]\n" \
  "buffer_load_dwordx4 %[A2], %[xrs], s65 offen offset:2048 lds\n" \
  "v_add_f32 v65, v65, v69\n" \
  "v_mfma_f32_16x16x32_bf16 v[40:43], a[72:75], v[48:51], v[40:43]\n" \
  "v_add_f32 v66, v66, v70\n" \
  "v_add_f32 v67, v67, v71\n" \
  "v_mfma_f32_16x16x32_bf16 v[44:47], a[76:79], v[48:51], v[44:47]\n" \
  "v_add_f32 v64, v64, v72\n" \
  "v_add_f32 v65, v65, v73\n" \
  "v_mfma_f32_16x16x32_bf16 v[32:35], a[80:83], v[52:55], v[32:35]\n" \
  "v_add_f32 v66, v66, v74\n" \
  "v_add_f32 v67, v67, v75\n" \
  "v_mfma_f32_16x16x32_bf16 v[36:39], a[84:87], v[52:55], v[36:39]\n" \
  "buffer_load_dwordx4 %[A2], %[xrs], s66 offen offset:3072 lds\n" \
  "v_add_f32 v64, v64, v76\n" \
  "v_mfma_f32_16x16x32_bf16 v[40:43], a[88:91], v[52:55], v[40:43]\n" \
  "s_add_u32 s65, s82, s89\n" \
  "v_add_f32 v65, v65, v77\n" \
  "v_mfma_f32_16x16x32_bf16 v[44:47], a[92:95], v[52:55], v[44:47]\n" \
  "s_waitcnt vmcnt(8) lgkmcnt(0)\n" \
  "ds_read_b128 v[48:51], %[vX0] offset:32768\n" \
  "v_mfma_f32_16x16x32_bf16 v[32:35], a[96:99], v[56:59], v[32:35]\n" \
  "ds_read_b128 v[52:55], %[vX1] offset:32768\n" \
  "s_add_u32 s66, s65, 64512\n" \
  "s_add_u32 m0, %[w4k], 16384\n" \
  "v_mfma_f32_16x16x32_bf16 v[36:39], a[100:103], v[56:59], v[36:39]\n" \
  "buffer_load_dwordx4 %[A1], %[xrs], s65 offen lds\n" \
  "v_add_f32 v66, v66, v78\n" \
  "v_mfma_f32_16x16x32_bf16 v[40:43], a[104:107], v[56:59], v[40:43]\n" \
  "v_add_f32 v67, v67, v79\n" \
  "v_mul_f32 v68, 0xbfb8aa3b, v64\n" \
  "v_mfma_f32_16x16x32_bf16 v[44:47], a[108:111], v[56:59], v[44:47]\n" \
  "ds_read_b128 v[56:59], %[vX0] offset:34816\n" \
  "v_mul_f32 v69, 0xbfb8aa3b, v65\n" \
  "v_mfma_f32_16x16x32_bf16 v[32:35], a[112:115], v[60:63], v[32:35]\n" \
  "v_mul_f32 v70, 0xbfb8aa3b, v66\n" \
  "v_mul_f32 v71, 0xbfb8aa3b, v67\n" \
  "v_mfma_f32_16x16x32_bf16 v[36:39], a[116:119], v[60:63], v[36:39]\n" \
  "buffer_load_dwordx4 %[A1], %[xrs], s66 offen offset:1024 lds\n" \
  "v_med3_f32 v68, v68, s73, -s73\n" \
  "v_mfma_f32_16x16x32_bf16 v[40:43], a[120:123], v[60:63], v[40:43]\n" \
  "v_med3_f32 v69, v69, s73, -s73\n" \
  "v_med3_f32 v70, v70, s73, -s73\n" \
  "v_mfma_f32_16x16x32_bf16 v[44:47], a[124:127], v[60:63], v[44:47]\n" \
  "ds_read_b128 v[60:63], %[vX1] offset:34816\n" \
  "v_med3_f32 v71, v71, s73, -s73\n" \
  "s_waitcnt lgkmcnt(3)\n" \
  "v_mfma_f32_16x16x32_bf16 v[32:35], a[128:131], v[48:51], v[32:35]\n" \
  "v_exp_f32 v68, v68\n" \
  "v_exp_f32 v69, v69\n" \
  "v_mfma_f32_16x16x32_bf16 v[36:39], a[132:135], v[48:51], v[36:39]\n" \
  "buffer_load_dwordx4 %[A2], %[xrs], s65 offen offset:2048 lds\n" \
  "v_exp_f32 v70, v70\n" \
  "v_mfma_f32_16x16x32_bf16 v[40:43], a[136:139], v[48:51], v[40:43]\n" \
  "v_exp_f32 v71, v71\n" \
  "v_add_f32 v68, 1.0, v68\n" \
  "v_mfma_f32_16x16x32_bf16 v[44:47], a[140:143], v[48:51], v[44:47]\n" \
  "v_add_f32 v69, 1.0, v69\n" \
  "v_add_f32 v70, 1.0, v70\n" \
  "s_waitcnt lgkmcnt(2)\n" \
  "v_mfma_f32_16x16x32_bf16 v[32:35], a[144:147], v[52:55], v[32:35]\n" \
  "v_add_f32 v71, 1.0, v71\n" \
  "v_rcp_f32 v68, v68\n" \
  "v_mfma_f32_16x16x32_bf16 v[36:39], a[148:151], v[52:55], v[36:39]\n" \
  "buffer_load_dwordx4 %[A2], %[xrs], s66 offen offset:3072 lds\n" \
  "v_rcp_f32 v69, v69\n" \
  "v_mfma_f32_16x16x32_bf16 v[40:43], a[152:155], v[52:55], v[40:43]\n" \
  "s_add_u32 s65, s82, s90\n" \
  "v_rcp_f32 v70, v70\n" \
  "v_mfma_f32_16x16x32_bf16 v[44:47], a[156:159], v[52:55], v[44:47]\n" \
  "s_waitcnt vmcnt(8) lgkmcnt(0)\n" \
  "ds_read_b128 v[48:51], %[vX0] offset:49152\n" \
  "v_mfma_f32_16x16x32_bf16 v[32:35], a[160:163], v[56:59], v[32:35]\n" \
  "ds_read_b128 v[52:55], %[vX1] offset:49152\n" \
  "s_add_u32 s66, s65, 64512\n" \
  "s_add_u32 m0, %[w4k], 32768\n" \
  "v_mfma_f32_16x16x32_bf16 v[36:39], a[164:167], v[56:59], v[36:39]\n" \
  "buffer_load_dwordx4 %[A1], %[xrs], s65 offen lds\n" \
  "v_rcp_f32 v71, v71\n" \
  "v_mfma_f32_16x16x32_bf16 v[40:43], a[168:171], v[56:59], v[40:43]\n" \
  "v_mul_f32 v64, v64, v68\n" \
  "v_mul_f32 v65, v65, v69\n" \
  "v_mfma_f32_16x16x32_bf16 v[44:47], a[172:175], v[56:59], v[44:47]\n" \
  "ds_read_b128 v[56:59], %[vX0] offset:51200\n" \
  "v_mul_f32 v66, v66, v70\n" \
  "v_mfma_f32_16x16x32_bf16 v[32:35], a[176:179], v[60:63], v[32:35]\n" \
  "v_mul_f32 v67, v67, v71\n" \
  "v_cvt_pk_bf16_f32 v20, v64, v65\n" \
  "v_mfma_f32_16x16x32_bf16 v[36:39], a[180:183], v[60:63], v[36:39]\n" \
  "buffer_load_dwordx4 %[A1], %[xrs], s66 offen offset:1024 lds\n" \
  "v_cvt_pk_bf16_f32 v21, v66, v67\n" \
  "v_mfma_f32_16x16x32_bf16 v[40:43], a[184:187], v[60:63], v[40:43]\n" \
  "v_mfma_f32_16x16x32_bf16 v[44:47], a[188:191], v[60:63], v[44:47]\n" \
  "ds_read_b128 v[60:63], %[vX1] offset:51200\n" \
  "s_waitcnt lgkmcnt(3)\n" \
  "v_mfma_f32_16x16x32_bf16 v[32:35], a[192:195], v[48:51], v[32:35]\n" \
  "v_mfma_f32_16x16x32_bf16 v[36:39], a[196:199], v[48:51], v[36:39]\n" \
  "buffer_load_dwordx4 %[A2], %[xrs], s65 offen offset:2048 lds\n" \
  "v_mfma_f32_16x16x32_bf16 v[40:43], a[200:203], v[48:51], v[40:43]\n" \
  "v_mfma_f32_16x16x32_bf16 v[44:47], a[204:207], v[48:51], v[44:47]\n" \
  "s_waitcnt lgkmcnt(2)\n" \
  "v_mfma_f32_16x16x32_bf16 v[32:35], a[208:211], v[52:55], v[32:35]\n" \
  "v_mfma_f32_16x16x32_bf16 v[36:39], a[212:215], v[52:55], v[36:39]\n" \
  "buffer_load_dwordx4 %[A2], %[xrs], s66 offen offset:3072 lds\n" \
  "v_mfma_f32_16x16x32_bf16 v[40:43], a[216:219], v[52:55], v[40:43]\n" \
  "s_add_u32 s65, s82, s91\n" \
  "v_mfma_f32_16x16x32_bf16 v[44:47], a[220:223], v[52:55], v[44:47]\n" \
  "s_waitcnt vmcnt(8) lgkmcnt(0)\n" \
  "ds_read_b128 v[48:51], %[vX0] offset:0\n" \
  "v_mfma_f32_16x16x32_bf16 v[32:35], a[224:227], v[56:59], v[32:35]\n" \
  "ds_read_b128 v[52:55], %[vX1] offset:0\n" \
  "s_add_u32 s66, s65, 64512\n" \
  "s_add_u32 m0, %[w4k], 49152\n" \
  "v_mfma_f32_16x16x32_bf16 v[36:39], a[228:231], v[56:59], v[36:39]\n" \
  "buffer_load_dwordx4 %[A1], %[xrs], s65 offen lds\n" \
  "v_mfma_f32_16x16x32_bf16 v[40:43], a[232:235], v[56:59], v[40:43]\n" \
  "v_mfma_f32_16x16x32_bf16 v[44:47], a[236:239], v[56:59], v[44:47]\n" \
  "ds_read_b128 v[56:59], %[vX0] offset:2048\n" \
  "v_mfma_f32_16x16x32_bf16 v[32:35], a[240:243], v[60:63], v[32:35]\n" \
  "v_mfma_f32_16x16x32_bf16 v[36:39], a[244:247], v[60:63], v[36:39]\n" \
  "buffer_load_dwordx4 %[A1], %[xrs], s66 offen offset:1024 lds\n" \
  "v_mfma_f32_16x16x32_bf16 v[40:43], a[248:251], v[60:63], v[40:43]\n" \
  "v_mfma_f32_16x16x32_bf16 v[44:47], a[252:255], v[60:63], v[44:47]\n" \
  "ds_read_b128 v[60:63], %[vX1] offset:2048\n" \
  "s_waitcnt lgkmcnt(3)\n" \
  "v_mfma_f32_16x16x32_bf16 v[32:35], v[80:83], v[48:51], v[32:35]\n" \
  "v_mfma_f32_16x16x32_bf16 v[36:39], v[84:87], v[48:51], v[36:39]\n" \
  "buffer_load_dwordx4 %[A2], %[xrs], s65 offen offset:2048 lds\n" \
  "ds_read_b128 v[64:67], %[vL16] offset:0\n" \
  "ds_read_b128 v[68:71], %[vL16] offset:1024\n" \
  "v_mfma_f32_16x16x32_bf16 v[40:43], v[88:91], v[48:51], v[40:43]\n" \
  "ds_read_b128 v[72:75], %[vL16] offset:2048\n" \
  "ds_read_b128 v[76:79], %[vL16] offset:3072\n" \
  "v_mfma_f32_16x16x32_bf16 v[44:47], v[92:95], v[48:51], v[44:47]\n" \
  "s_waitcnt lgkmcnt(6)\n" \
  "v_mfma_f32_16x16x32_bf16 v[32:35], v[96:99], v[52:55], v[32:35]\n" \
  "s_add_u32 s82, s82, 131072\n" \
  "v_mfma_f32_16x16x32_bf16 v[36:39], v[100:103], v[52:55], v[36:39]\n" \
  "buffer_load_dwordx4 %[A2], %[xrs], s66 offen offset:3072 lds\n" \
  "v_mfma_f32_16x16x32_bf16 v[40:43], v[104:107], v[52:55], v[40:43]\n" \
  "s_add_u32 s65, s82, s84\n" \
  "v_mfma_f32_16x16x32_bf16 v[44:47], v[108:111], v[52:55], v[44:47]\n" \
  "s_waitcnt vmcnt(8) lgkmcnt(4)\n" \
  "ds_read_b128 v[48:51], %[vX0] offset:16384\n" \
  "s_barrier\n" \
  "s_waitcnt lgkmcnt(1)\n" \
  "v_mfma_f32_16x16x32_bf16 v[32:35], v[64:67], v[56:59], v[32:35]\n" \
  "ds_read_b128 v[52:55], %[vX1] offset:16384\n" \
  "s_add_u32 s66, s65, 64512\n" \
  "s_add_u32 m0, %[w4k], 0\n" \
  "v_mfma_f32_16x16x32_bf16 v[36:39], v[68:71], v[56:59], v[36:39]\n" \
  "buffer_load_dwordx4 %[A1], %[xrs], s65 offen lds\n" \
  "v_mfma_f32_16x16x32_bf16 v[40:43], v[72:75], v[56:59], v[40:43]\n" \
  "v_mfma_f32_16x16x32_bf16 v[44:47], v[76:79], v[56:59], v[44:47]\n" \
  "ds_read_b128 v[56:59], %[vX0] offset:18432\n" \
  "v_mfma_f32_16x16x32_bf16 v[32:35], v[112:115], v[60:63], v[32:35]\n" \
  "v_mfma_f32_16x16x32_bf16 v[36:39], v[116:119], v[60:63], v[36:39]\n" \
  "buffer_load_dwordx4 %[A1], %[xrs], s66 offen offset:1024 lds\n" \
  "ds_read_b128 v[64:67], %[vL16] offset:16384\n" \
  "ds_read_b128 v[68:71], %[vL16] offset:17408\n" \
  "v_mfma_f32_16x16x32_bf16 v[40:43], v[120:123], v[60:63], v[40:43]\n" \
  "ds_read_b128 v[72:75], %[vL16] offset:18432\n" \
  "ds_read_b128 v[76:79], %[vL16] offset:19456\n" \
  "v_mfma_f32_16x16x32_bf16 v[44:47], v[124:127], v[60:63], v[44:47]\n" \
  "ds_read_b128 v[60:63], %[vX1] offset:18432\n" \
  "s_waitcnt lgkmcnt(7)\n" \
  "v_mfma_f32_16x16x32_bf16 v[32:35], v[128:131], v[48:51], v[32:35]\n" \
  "v_mfma_f32_16x16x32_bf16 v[36:39], v[132:135], v[48:51], v[36:39]\n" \
  "buffer_load_dwordx4 %[A2], %[xrs], s65 offen offset:2048 lds\n" \
  "v_mfma_f32_16x16x32_bf16 v[40:43], v[136:139], v[48:51], v[40:43]\n" \
  "v_mfma_f32_16x16x32_bf16 v[44:47], v[140:143], v[48:51], v[44:47]\n" \
  "s_waitcnt lgkmcnt(1)\n" \
  "v_mfma_f32_16x16x32_bf16 v[32:35], v[64:67], v[52:55], v[32:35]\n" \
  "v_mfma_f32_16x16x32_bf16 v[36:39], v[68:71], v[52:55], v[36:39]\n" \
  "buffer_load_dwordx4 %[A2], %[xrs], s66 offen offset:3072 lds\n" \
  "v_mfma_f32_16x16x32_bf16 v[40:43], v[72:75], v[52:55], v[40:43]\n" \
  "s_add_u32 s65, s82, s85\n" \
  "v_mfma_f32_16x16x32_bf16 v[44:47], v[76:79], v[52:55], v[44:47]\n" \
  "s_waitcnt vmcnt(8) lgkmcnt(0)\n" \
  "ds_read_b128 v[48:51], %[vX0] offset:32768\n" \
  "v_mfma_f32_16x16x32_bf16 v[32:35], v[144:147], v[56:59], v[32:35]\n" \
  "ds_read_b128 v[52:55], %[vX1] offset:32768\n" \
  "s_add_u32 s66, s65, 64512\n" \
  "s_add_u32 m0, %[w4k], 16384\n" \
  "v_mfma_f32_16x16x32_bf16 v[36:39], v[148:151], v[56:59], v[36:39]\n" \
  "buffer_load_dwordx4 %[A1], %[xrs], s65 offen lds\n" \
  "ds_read_b128 v[64:67], %[vL16] offset:32768\n" \
  "ds_read_b128 v[68:71], %[vL16] offset:33792\n" \
  "v_mfma_f32_16x16x32_bf16 v[40:43], v[152:155], v[56:59], v[40:43]\n" \
  "ds_read_b128 v[72:75], %[vL16] offset:34816\n" \
  "ds_read_b128 v[76:79], %[vL16] offset:35840\n" \
  "v_mfma_f32_16x16x32_bf16 v[44:47], v[156:159], v[56:59], v[44:47]\n" \
  "ds_read_b128 v[56:59], %[vX0] offset:34816\n" \
  "v_mfma_f32_16x16x32_bf16 v[32:35], v[160:163], v[60:63], v[32:35]\n" \
  "v_mfma_f32_16x16x32_bf16 v[36:39], v[164:167], v[60:63], v[36:39]\n" \
  "buffer_load_dwordx4 %[A1], %[xrs], s66 offen offset:1024 lds\n" \
  "v_mfma_f32_16x16x32_bf16 v[40:43], v[168:171], v[60:63], v[40:43]\n" \
  "v_mfma_f32_16x16x32_bf16 v[44:47], v[172:175], v[60:63], v[44:47]\n" \
  "ds_read_b128 v[60:63], %[vX1] offset:34816\n" \
  "buffer_store_dwordx2 v[20:21], %[VG], s[92:95], s71 offen sc0 sc1\n" \
  "s_waitcnt lgkmcnt(2)\n" \
  "v_mfma_f32_16x16x32_bf16 v[32:35], v[64:67], v[48:51], v[32:35]\n" \
  "s_add_u32 s71, s71, 16384\n" \
  "v_mfma_f32_16x16x32_bf16 v[36:39], v[68:71], v[48:51], v[36:39]\n" \
  "buffer_load_dwordx4 %[A2], %[xrs], s65 offen offset:2048 lds\n" \
  "v_mfma_f32_16x16x32_bf16 v[40:43], v[72:75], v[48:51], v[40:43]\n" \
  "v_mfma_f32_16x16x32_bf16 v[44:47], v[76:79], v[48:51], v[44:47]\n" \
  "v_mfma_f32_16x16x32_bf16 v[32:35], v[176:179], v[52:55], v[32:35]\n" \
  "v_mfma_f32_16x16x32_bf16 v[36:39], v[180:183], v[52:55], v[36:39]\n" \
  "buffer_load_dwordx4 %[A2], %[xrs], s66 offen offset:3072 lds\n" \
  "ds_read_b128 v[64:67], %[vL16] offset:49152\n" \
  "ds_read_b128 v[68:71], %[vL16] offset:50176\n" \
  "v_mfma_f32_16x16x32_bf16 v[40:43], v[184:187], v[52:55], v[40:43]\n" \
  "s_add_u32 s65, s82, s86\n" \
  "ds_read_b128 v[72:75], %[vL16] offset:51200\n" \
  "ds_read_b128 v[76:79], %[vL16] offset:52224\n" \
  "v_mfma_f32_16x16x32_bf16 v[44:47], v[188:191], v[52:55], v[44:47]\n" \
  "s_waitcnt vmcnt(9) lgkmcnt(4)\n" \
  "ds_read_b128 v[48:51], %[vX0] offset:49152\n" \
  "v_mfma_f32_16x16x32_bf16 v[32:35], v[192:195], v[56:59], v[32:35]\n" \
  "ds_read_b128 v[52:55], %[vX1] offset:49152\n" \
  "s_add_u32 s66, s65, 64512\n" \
  "s_add_u32 m0, %[w4k], 32768\n" \
  "v_mfma_f32_16x16x32_bf16 v[36:39], v[196:199], v[56:59], v[36:39]\n" \
  "buffer_load_dwordx4 %[A1], %[xrs], s65 offen lds\n" \
  "v_mfma_f32_16x16x32_bf16 v[40:43], v[200:203], v[56:59], v[40:43]\n" \
  "v_mfma_f32_16x16x32_bf16 v[44:47], v[204:207], v[56:59], v[44:47]\n" \
  "ds_read_b128 v[56:59], %[vX0] offset:51200\n" \
  "s_waitcnt lgkmcnt(3)\n" \
  "v_mfma_f32_16x16x32_bf16 v[32:35], v[64:67], v[60:63], v[32:35]\n" \
  "v_mfma_f32_16x16x32_bf16 v[36:39], v[68:71], v[60:63], v[36:39]\n" \
  "buffer_load_dwordx4 %[A1], %[xrs], s66 offen offset:1024 lds\n" \
  "v_mfma_f32_16x16x32_bf16 v[40:43], v[72:75], v[60:63], v[40:43]\n" \
  "v_mfma_f32_16x16x32_bf16 v[44:47], v[76:79], v[60:63], v[44:47]\n" \
  "ds_read_b128 v[60:63], %[vX1] offset:51200\n" \
  "s_waitcnt lgkmcnt(3)\n" \
  "v_mfma_f32_16x16x32_bf16 v[32:35], v[208:211], v[48:51], v[32:35]\n" \
  "v_mfma_f32_16x16x32_bf16 v[36:39], v[212:215], v[48:51], v[36:39]\n" \
  "buffer_load_dwordx4 %[A2], %[xrs], s65 offen offset:2048 lds\n" \
  "ds_read_b128 v[64:67], %[vL16b] offset:0\n" \
  "ds_read_b128 v[68:71], %[vL16b] offset:1024\n" \
  "v_mfma_f32_16x16x32_bf16 v[40:43], v[216:219], v[48:51], v[40:43]\n" \
  "ds_read_b128 v[72:75], %[vL16b] offset:2048\n" \
  "ds_read_b128 v[76:79], %[vL16b] offset:3072\n" \
  "v_mfma_f32_16x16x32_bf16 v[44:47], v[220:223], v[48:51], v[44:47]\n" \
  "s_waitcnt lgkmcnt(6)\n" \
  "v_mfma_f32_16x16x32_bf16 v[32:35], v[224:227], v[52:55], v[32:35]\n" \
  "v_mfma_f32_16x16x32_bf16 v[36:39], v[228:231], v[52:55], v[36:39]\n" \
  "buffer_load_dwordx4 %[A2], %[xrs], s66 offen offset:3072 lds\n" \
  "v_mfma_f32_16x16x32_bf16 v[40:43], v[232:235], v[52:55], v[40:43]\n" \
  "s_add_u32 s65, s82, s87\n" \
  "v_mfma_f32_16x16x32_bf16 v[44:47], v[236:239], v[52:55], v[44:47]\n" \
  "s_waitcnt vmcnt(9) lgkmcnt(4)\n" \
  "ds_read_b128 v[48:51], %[vX0] offset:0\n" \
  "s_waitcnt lgkmcnt(1)\n" \
  "v_mfma_f32_16x16x32_bf16 v[32:35], v[64:67], v[56:59], v[32:35]\n" \
  "ds_read_b128 v[52:55], %[vX1] offset:0\n" \
  "s_add_u32 s66, s65, 64512\n" \
  "s_add_u32 m0, %[w4k], 49152\n" \
  "v_mfma_f32_16x16x32_bf16 v[36:39], v[68:71], v[56:59], v[36:39]\n" \
  "buffer_load_dwordx4 %[A1], %[xrs], s65 offen lds\n" \
  "v_mfma_f32_16x16x32_bf16 v[40:43], v[72:75], v[56:59], v[40:43]\n" \
  "v_mfma_f32_16x16x32_bf16 v[44:47], v[76:79], v[56:59], v[44:47]\n" \
  "ds_read_b128 v[56:59], %[vX0] offset:2048\n" \
  "v_mfma_f32_16x16x32_bf16 v[32:35], v[240:243], v[60:63], v[32:35]\n" \
  "s_sub_u32 s64, s64, 1\n" \
  "v_mfma_f32_16x16x32_bf16 v[36:39], v[244:247], v[60:63], v[36:39]\n" \
  "buffer_load_dwordx4 %[A1], %[xrs], s66 offen offset:1024 lds\n" \
  "v_mfma_f32_16x16x32_bf16 v[40:43], v[248:251], v[60:63], v[40:43]\n" \
  "v_mfma_f32_16x16x32_bf16 v[44:47], v[252:255], v[60:63], v[44:47]\n" \
  "ds_read_b128 v[60:63], %[vX1] offset:2048\n" \
  "s_cmp_lg_u32 s64, 0\n" \
  "s_cbranch_scc1 L_loop_%=\n" \
  "s_nop 7\n" \
  "ds_write_b128 %[vPw0], v[32:35] offset:0\n" \
  "ds_write_b128 %[vPw1], v[36:39] offset:0\n" \
  "ds_write_b128 %[vPw0], v[40:43] offset:128\n" \
  "ds_write_b128 %[vPw1], v[44:47] offset:128\n" \
  "s_waitcnt lgkmcnt(0)\n" \
  "s_barrier\n" \
  "ds_read_b128 v[64:67], %[vPr] offset:0\n" \
  "ds_read_b128 v[68:71], %[vPr] offset:4096\n" \
  "ds_read_b128 v[72:75], %[vPr] offset:8192\n" \
  "ds_read_b128 v[76:79], %[vPr] offset:12288\n" \
  "s_waitcnt lgkmcnt(0)\n" \
  "v_add_f32 v64, v64, v68\n" \
  "v_add_f32 v65, v65, v69\n" \
  "v_add_f32 v66, v66, v70\n" \
  "v_add_f32 v67, v67, v71\n" \
  "v_add_f32 v64, v64, v72\n" \
  "v_add_f32 v65, v65, v73\n" \
  "v_add_f32 v66, v66, v74\n" \
  "v_add_f32 v67, v67, v75\n" \
  "v_add_f32 v64, v64, v76\n" \
  "v_add_f32 v65, v65, v77\n" \
  "v_add_f32 v66, v66, v78\n" \
  "v_add_f32 v67, v67, v79\n" \
  "v_mul_f32 v68, 0xbfb8aa3b, v64\n" \
  "v_mul_f32 v69, 0xbfb8aa3b, v65\n" \
  "v_mul_f32 v70, 0xbfb8aa3b, v66\n" \
  "v_mul_f32 v71, 0xbfb8aa3b, v67\n" \
  "v_med3_f32 v68, v68, s73, -s73\n" \
  "v_med3_f32 v69, v69, s73, -s73\n" \
  "v_med3_f32 v70, v70, s73, -s73\n" \
  "v_med3_f32 v71, v71, s73, -s73\n" \
  "v_exp_f32 v68, v68\n" \
  "v_exp_f32 v69, v69\n" \
  "v_exp_f32 v70, v70\n" \
  "v_exp_f32 v71, v71\n" \
  "v_add_f32 v68, 1.0, v68\n" \
  "v_add_f32 v69, 1.0, v69\n" \
  "v_add_f32 v70, 1.0, v70\n" \
  "v_add_f32 v71, 1.0, v71\n" \
  "v_rcp_f32 v68, v68\n" \
  "v_rcp_f32 v69, v69\n" \
  "v_rcp_f32 v70, v70\n" \
  "v_rcp_f32 v71, v71\n" \
  "v_mul_f32 v64, v64, v68\n" \
  "v_mul_f32 v65, v65, v69\n" \
  "v_mul_f32 v66, v66, v70\n" \
  "v_mul_f32 v67, v67, v71\n" \
  "v_cvt_pk_bf16_f32 v20, v64, v65\n" \
  "v_cvt_pk_bf16_f32 v21, v66, v67\n" \
  "buffer_store_dwordx2 v[20:21], %[VG], s[92:95], s71 offen sc0 sc1\n"

constexpr int NR = 8;
constexpr int MLOC = 512;
constexpr int KD = 4096;
constexpr int ND = 4096;
constexpr int NLOC = 512;
constexpr int LDS_BYTES = 163840;
constexpr int NWG = 256;

constexpr size_t WS_FLAGS = 0;
constexpr size_t WS_CNT = 4096;
constexpr size_t WS_TMO = 8192;
constexpr size_t WS_NEED = 16384;
constexpr unsigned long long DEADLINE_TICKS = 20000000ull;

struct Params {
  const u16* x; const u16* w;
  u16* out[NR]; unsigned* ws[NR];
  int rank; int nranks;
};

#define V10(p) p "0", p "1", p "2", p "3", p "4", p "5", p "6", p "7", p "8", p "9"
#define CLOB_V V10("v2"), V10("v3"), V10("v4"), V10("v5"), V10("v6"), V10("v7"), V10("v8"), V10("v9"), \
  V10("v10"), V10("v11"), V10("v12"), V10("v13"), V10("v14"), V10("v15"), V10("v16"), V10("v17"), V10("v18"), V10("v19"), \
  V10("v20"), V10("v21"), V10("v22"), V10("v23"), V10("v24"), "v250", "v251", "v252", "v253", "v254", "v255"
#define CLOB_A "a0", "a1", "a2", "a3", "a4", "a5", "a6", "a7", "a8", "a9", V10("a1"), V10("a2"), V10("a3"), V10("a4"), V10("a5"), V10("a6"), V10("a7"), V10("a8"), V10("a9"), \
  V10("a10"), V10("a11"), V10("a12"), V10("a13"), V10("a14"), V10("a15"), V10("a16"), V10("a17"), V10("a18"), V10("a19"), \
  V10("a20"), V10("a21"), V10("a22"), V10("a23"), V10("a24"), "a250", "a251", "a252", "a253", "a254", "a255"
#define CLOB_S "s64", "s65", "s66", "s67", "s68", "s69", V10("s7"), V10("s8"), "s90", "s91", "s92", "s93", "s94", "s95"

__global__ __launch_bounds__(256, 1) void gemm_a2a_kernel(Params p) {
#if defined(__HIP_DEVICE_COMPILE__)
  const int bid = blockIdx.x;
  const int dest = bid & 7, li = bid >> 3;
  const int rg = li & 3, cg = li >> 2;
  const int row0 = rg * 128, col0 = dest * NLOC + cg * 64;
  const int lane = threadIdx.x & 63;
  const int wid = __builtin_amdgcn_readfirstlane(threadIdx.x >> 6);

  auto xrs = __builtin_amdgcn_make_buffer_rsrc((void*)((const char*)p.x - 2048), 0, (row0 + 128) * (KD * 2) + 2048, 0x00020000);
  auto wrs = __builtin_amdgcn_make_buffer_rsrc((void*)p.w, 0, KD * ND * 2, 0x00020000);
  const char* optr = (const char*)__builtin_amdgcn_kernarg_segment_ptr() + offsetof(Params, out) + dest * 8;

  unsigned perm = 0;
  {
    const int fa = (cg & 1) * 4, fb = fa ^ 4;
    const int r1 = rg + (cg >> 1), r0 = (cg & 1) ? r1 : (cg >> 1);
    for (int g = 0; g < 4; ++g)
      perm |= ((unsigned)(fa | ((g + r0) & 3)) << (4 * g)) | ((unsigned)(fb | ((g + r1) & 3)) << (4 * g + 16));
  }
  const unsigned w4k = wid * 4096;
  const unsigned xrow = row0 * (KD * 2);
  const unsigned obase = (p.rank * MLOC + row0) * (NLOC * 2) + cg * 128;

  const int l3 = lane >> 3, l7 = lane & 7, m = lane & 15, q = lane >> 4;
  const unsigned xl = l3 * (KD * 2) + ((l7 ^ (2 * ((l3 >> 1) & 3))) * 16) + 2048;
  const unsigned A1 = xl + wid * 256, A2 = A1 + 128 - 2048;
  const int a = (m < 4) ? m : (m < 12 ? m + 4 : m - 8);
  const unsigned vX0 = w4k + a * 128 + ((q ^ (2 * ((a >> 1) & 3))) * 16), vX1 = vX0 ^ 64;
  const int hw = ((l3 >> 1) & 1) | (((l3 >> 2) & 1) << 1);
  const unsigned A3 = (unsigned)(wid * 128 + 8 * (l3 >> 2) + (l3 & 3)) * (ND * 2) + col0 * 2 + ((l7 ^ (hw << 1)) * 16);
  const int tq = (lane >> 2) & 3, tp = lane & 3, th = ((tq >> 1) & 1) | ((q & 1) << 1);
  const unsigned tb = w4k + (2 * (q >> 1)) * 1024 + (4 * (q & 1) + tq) * 128 + (tp >> 1) * 16 + (tp & 1) * 8;
  const unsigned vT0 = tb + ((0 ^ th) * 32), vT1 = tb + ((1 ^ th) * 32), vT2 = tb + ((2 ^ th) * 32), vT3 = tb + ((3 ^ th) * 32);
  const unsigned vL16 = 65536 + w4k + lane * 16, vL16b = vL16 + 65536;
  const unsigned vPw0 = 147456 + w4k + a * 256 + ((q ^ (a & 7)) * 16), vPw1 = vPw0 ^ 64;
  const int pr = 4 * wid + q;
  const unsigned vPr = 147456 + pr * 256 + ((m ^ (pr & 7)) * 16);
  const unsigned VG = pr * (NLOC * 2) + m * 8;

  asm volatile(S2_ASM
               :
               : [w4k] "s"(w4k), [xrow] "s"(xrow), [obase] "s"(obase), [xrs] "s"(xrs), [wrs] "s"(wrs), [optr] "s"(optr), [perm] "s"(perm),
                 [vX0] "v"(vX0), [vX1] "v"(vX1), [vT0] "v"(vT0), [vT1] "v"(vT1), [vT2] "v"(vT2), [vT3] "v"(vT3),
                 [vL16] "v"(vL16), [vL16b] "v"(vL16b), [vPw0] "v"(vPw0), [vPw1] "v"(vPw1), [vPr] "v"(vPr), [VG] "v"(VG),
                 [A1] "v"(A1), [A2] "v"(A2), [A3] "v"(A3)
               : CLOB_V, CLOB_A, CLOB_S, "scc", "vcc", "memory");

  unsigned* myws = p.ws[p.rank];
  unsigned call = 0;
  if (wid == 0) call = __hip_atomic_load(myws + WS_CNT / 4 + bid, __ATOMIC_RELAXED, __HIP_MEMORY_SCOPE_AGENT) + 1;
  asm volatile("s_waitcnt vmcnt(0)" ::: "memory");
  __builtin_amdgcn_s_barrier();
  asm volatile("" ::: "memory");
  if (wid == 0) {
    call = __builtin_amdgcn_readfirstlane(call);
    const int ln = __builtin_amdgcn_mbcnt_hi(~0u, __builtin_amdgcn_mbcnt_lo(~0u, 0u));
    if (ln == 0) {
      __hip_atomic_store(p.ws[dest] + WS_FLAGS / 4 + p.rank * 32 + li, call, __ATOMIC_RELAXED, __HIP_MEMORY_SCOPE_SYSTEM);
      __hip_atomic_store(myws + WS_CNT / 4 + bid, call, __ATOMIC_RELAXED, __HIP_MEMORY_SCOPE_AGENT);
    }
    if (bid == 0) {
      unsigned* fl = myws + WS_FLAGS / 4 + ln * 4;
      const unsigned long long t0 = __builtin_amdgcn_s_memrealtime();
      bool bad = false;
      u32x4 fa, fb;
      asm volatile("global_load_dwordx4 %0, %1, off sc0 sc1" : "=&v"(fa) : "v"(fl) : "memory");
      for (;;) {
        asm volatile("global_load_dwordx4 %1, %2, off sc0 sc1\n\ts_waitcnt vmcnt(1)" : "+v"(fa), "=&v"(fb) : "v"(fl) : "memory");
        if (__all((fa.x >= call) & (fa.y >= call) & (fa.z >= call) & (fa.w >= call))) break;
        asm volatile("global_load_dwordx4 %0, %2, off sc0 sc1\n\ts_waitcnt vmcnt(1)" : "=&v"(fa), "+v"(fb) : "v"(fl) : "memory");
        if (__all((fb.x >= call) & (fb.y >= call) & (fb.z >= call) & (fb.w >= call))) break;
        if (__builtin_amdgcn_s_memrealtime() - t0 > DEADLINE_TICKS) { bad = true; break; }
      }
      asm volatile("s_waitcnt vmcnt(0)" : "+v"(fa), "+v"(fb) :: "memory");
      if (bad && ln == 0) {
        __hip_atomic_store(myws + WS_TMO / 4, call, __ATOMIC_RELAXED, __HIP_MEMORY_SCOPE_AGENT);
        u16* o = p.out[p.rank] + (size_t)(p.rank * MLOC) * NLOC;
        __hip_atomic_store(o, (u16)0x7fc0, __ATOMIC_RELAXED, __HIP_MEMORY_SCOPE_SYSTEM);
      }
    }
  }
#endif
}

extern "C" void kernel_launch(int rank, int nranks, ncclComm_t comm, hipStream_t stream,
                              void* const* d_in, const int* in_sizes, int n_in,
                              void* d_out, int out_size, const int* in_dtypes, int out_dtype,
                              void* const* d_out_all, void* const* d_ws_all, size_t ws_bytes) {
  if (nranks != NR || rank < 0 || rank >= NR || n_in < 2 || in_sizes[0] != MLOC * KD || in_sizes[1] != KD * ND ||
      out_size != ND * NLOC || ws_bytes < WS_NEED) {
    fprintf(stderr, "unexpected problem shape\n");
    return;
  }
  static bool init = false;
  if (!init) {
    hipError_t e = hipFuncSetAttribute((const void*)gemm_a2a_kernel, hipFuncAttributeMaxDynamicSharedMemorySize, LDS_BYTES);
    if (e != hipSuccess) { fprintf(stderr, "hipFuncSetAttribute: %s\n", hipGetErrorString(e)); return; }
    init = true;
  }
  Params p;
  p.x = (const u16*)d_in[0]; p.w = (const u16*)d_in[1];
  for (int i = 0; i < NR; ++i) { p.out[i] = (u16*)d_out_all[i]; p.ws[i] = (unsigned*)d_ws_all[i]; }
  p.rank = rank; p.nranks = nranks;
  gemm_a2a_kernel<<<NWG, 256, LDS_BYTES, stream>>>(p);
}
```
